```python
import numpy as np

kernel_name = "gpu_ag_gemm_m4096_k4096_n4096_e5m2e5m2_dqsilu"


ABSMAX_RTOL = 0.05

COMPUTE_DTYPE = "float32"
EPILOGUE = "dequant_silu"
IN_QUANT = None
OUT_QUANT = None
IN_FP8 = "e5m2e5m2"
IN_DTYPES = {"x": "e5m2", "w_mat": "e5m2",
             "scale_x": "float32", "scale_w": "float32"}
WORLD_SIZE = 8

def _snap_fp8(x, mant_bits, max_abs, min_normal):
    a = np.abs(x.astype(np.float64))
    _m, _e = np.frexp(a)
    q = np.where(a >= min_normal, np.ldexp(1.0, _e - 1 - mant_bits),
                 min_normal * 2.0 ** -mant_bits)
    snapped = np.minimum(np.round(a / q) * q, max_abs)
    return (np.sign(x) * snapped).astype(np.float32)

def setup_inputs():

    x = _snap_fp8(np.random.randn(4096, 4096).astype(np.float32), 2, 57344.0, 2.0 ** -14)
    w_mat = _snap_fp8(np.random.randn(4096, 4096).astype(np.float32), 2, 57344.0, 2.0 ** -14)
    scale_x = (0.02 * (1.0 + np.random.rand(1))).astype(np.float32)
    scale_w = (0.02 * (1.0 + np.random.rand(1))).astype(np.float32)
    return {"x": x, "w_mat": w_mat, "scale_x": scale_x, "scale_w": scale_w}

def reference(x, w_mat, scale_x, scale_w):
    acc = (x @ w_mat).astype(np.float32)
    y = (acc * (scale_x * scale_w)).astype(np.float32)
    return (y / (1.0 + np.exp(-np.clip(y, -60.0, 60.0)))).astype(np.float32)
```

```cpp
#include <hip/hip_runtime.h>
#include <rccl/rccl.h>
#include <cstdio>
#include <cstdint>

#define HIP_CHECK(expr) do { hipError_t _e = (expr); if (_e != hipSuccess) { \
    fprintf(stderr, "HIP error %s at %s:%d\n", hipGetErrorString(_e), __FILE__, __LINE__); \
    return; } } while (0)

namespace {

constexpr int NR = 8;
constexpr int KDIM = 4096;
constexpr int MLOC = 512;
constexpr int NLOC = 512;
constexpr unsigned SHARD = MLOC * KDIM;
constexpr unsigned UNIT_BYTES = 32 * KDIM;
constexpr int NCHUNK = 16;
constexpr unsigned CHUNK_BYTES = 8192;
constexpr size_t STAGE_BYTES = (size_t)NR * SHARD;
constexpr size_t FLAG_OFF = 2 * STAGE_BYTES;
constexpr size_t CNT_OFF = FLAG_OFF + 8 * 16 * 128;
constexpr size_t TMO_OFF = CNT_OFF + 256 * 16 * 4;
constexpr size_t WS_NEED = TMO_OFF + 128;
constexpr unsigned long long DEADLINE_TICKS = 40000000ull;
constexpr int LDS_W = KDIM * 32;
constexpr int AUX_SYS = 17;

typedef unsigned int u32x4 __attribute__((ext_vector_type(4)));
typedef int i32x2 __attribute__((ext_vector_type(2)));
typedef int i32x8 __attribute__((ext_vector_type(8)));
typedef float f32x4 __attribute__((ext_vector_type(4)));
typedef __attribute__((address_space(3))) i32x2* lds_i32x2_ptr;
typedef __attribute__((address_space(3))) volatile unsigned* lds_u32_ptr;

struct Params {
    const unsigned char* x;
    const unsigned char* w;
    const float* sx;
    const float* sw;
    float* out;
    unsigned char* ws[NR];
    int rank;
};

__device__ __forceinline__ unsigned ld_sys(const unsigned* p) {
    return __hip_atomic_load(p, __ATOMIC_RELAXED, __HIP_MEMORY_SCOPE_SYSTEM);
}
__device__ __forceinline__ void st_sys(unsigned* p, unsigned v) {
    __hip_atomic_store(p, v, __ATOMIC_RELAXED, __HIP_MEMORY_SCOPE_SYSTEM);
}
__device__ __forceinline__ unsigned ld_agent(const unsigned* p) {
    return __hip_atomic_load(p, __ATOMIC_RELAXED, __HIP_MEMORY_SCOPE_AGENT);
}
__device__ __forceinline__ void st_agent(unsigned* p, unsigned v) {
    __hip_atomic_store(p, v, __ATOMIC_RELAXED, __HIP_MEMORY_SCOPE_AGENT);
}
__device__ __forceinline__ unsigned rfl(unsigned v) { return __builtin_amdgcn_readfirstlane(v); }

__device__ __forceinline__ i32x8 pack8(u32x4 lo, u32x4 hi) {
    i32x8 r;
    r[0] = lo[0]; r[1] = lo[1]; r[2] = lo[2]; r[3] = lo[3];
    r[4] = hi[0]; r[5] = hi[1]; r[6] = hi[2]; r[7] = hi[3];
    return r;
}

__global__ void __launch_bounds__(576)
ag_gemm_silu_kernel(Params p) {
    __shared__ __attribute__((aligned(16))) unsigned char lds[LDS_W + 64];
    const int lane = threadIdx.x & 63;
    const int wave = rfl(threadIdx.x >> 6);
    const int bid = blockIdx.x;
    const int label = bid & 7;
    const int idx = bid >> 3;
    unsigned char* const myws = p.ws[p.rank & 7];
    lds_u32_ptr sync = (lds_u32_ptr)(lds + LDS_W);

    if (wave == 0) {
        if (label == 0 || idx >= 8) { __builtin_amdgcn_s_barrier(); return; }
        unsigned* cnt = (unsigned*)(myws + CNT_OFF) + bid * 16;
        const unsigned cv = ld_agent(cnt);
        const auto xr = __builtin_amdgcn_make_buffer_rsrc((void*)p.x, 0, SHARD, 0x00020000);
        const unsigned vx0 = (lane & 15) * KDIM + (lane >> 4) * 16;
        const unsigned vx1 = vx0 + 16 * KDIM;
        u32x4 d[2][8];
        auto load_burst = [&](u32x4 (&r)[8], int b) {
            const unsigned so = (unsigned)(idx + 8 * ((b >> 4) & 1)) * UNIT_BYTES + (unsigned)(b & 15) * 256;
#pragma unroll
            for (int s = 0; s < 2; ++s)
#pragma unroll
                for (int ms = 0; ms < 2; ++ms)
#pragma unroll
                    for (int h = 0; h < 2; ++h)
                        r[(s * 2 + ms) * 2 + h] =
                            __builtin_amdgcn_raw_buffer_load_b128(xr, (ms ? vx1 : vx0) + s * 128 + h * 64, so, 0);
        };
        load_burst(d[0], 0);
        load_burst(d[1], 1);
        __builtin_amdgcn_s_barrier();
        const unsigned call = rfl(cv) + 1;
        if (lane == 0) st_agent(cnt, call);
        const unsigned dst = (unsigned)(p.rank + label) & 7;
        unsigned char* const dws = p.ws[dst];
        const auto dr = __builtin_amdgcn_make_buffer_rsrc(
            dws + (size_t)(call & 1) * STAGE_BYTES + (size_t)(p.rank & 7) * SHARD, 0, SHARD, 0x00020000);
        unsigned* const flags = (unsigned*)(dws + FLAG_OFF) + (p.rank & 7) * 16 * 32;
        const unsigned fbase = (call - 1) * NCHUNK;
        const unsigned vs = lane * 16;
        auto store_burst = [&](const u32x4 (&r)[8], int b) {
            const unsigned so = (unsigned)(idx + 8 * (b >> 4)) * UNIT_BYTES + (unsigned)(b & 15) * CHUNK_BYTES;
#pragma unroll
            for (int i = 0; i < 8; ++i)
                __builtin_amdgcn_raw_buffer_store_b128(r[i], dr, vs + (i & 3) * 1024, so + (i >> 2) * 4096, AUX_SYS);
        };
        auto flag = [&](int b) {
            if (lane == 0) st_sys(flags + (idx + 8 * (b >> 4)) * 32, fbase + (b & 15) + 1);
        };
        store_burst(d[0], 0);
        load_burst(d[0], 2);
        store_burst(d[1], 1);
        load_burst(d[1], 3);
        asm volatile("s_waitcnt vmcnt(24)" ::: "memory");
        flag(0);
        for (int b = 2; b < 32; b += 2) {
            store_burst(d[0], b);
            load_burst(d[0], b + 2);
            asm volatile("s_waitcnt vmcnt(25)" ::: "memory");
            flag(b - 1);
            store_burst(d[1], b + 1);
            load_burst(d[1], b + 3);
            asm volatile("s_waitcnt vmcnt(25)" ::: "memory");
            flag(b);
        }
        asm volatile("s_waitcnt vmcnt(0)" ::: "memory");
        flag(31);
        return;
    }

    const int cw = wave - 1;
    const int nblk = idx & 15, rg = idx >> 4;
    const int unit = 8 * (cw >> 2) + (cw & 3) * 2 + rg;
    const unsigned src = (unsigned)(p.rank - label) & 7;
    unsigned* cnt = (unsigned*)(myws + CNT_OFF) + bid * 16 + wave;
    const unsigned cv = ld_agent(cnt);
    if (cw == 0 && lane < 8) sync[lane] = 0;
    __builtin_amdgcn_s_waitcnt(0xC07F);
    __builtin_amdgcn_s_barrier();

    {
        const unsigned char* wsrc = p.w + (size_t)(lane >> 1) * NLOC + nblk * 32 + (((lane & 1) ^ ((lane >> 5) & 1)) * 16);
#pragma unroll
        for (int i = 0; i < 16; ++i) {
            const int blk = i * 8 + cw;
            __builtin_amdgcn_global_load_lds((const __attribute__((address_space(1))) void*)(wsrc + (size_t)blk * 32 * NLOC),
                                             (__attribute__((address_space(3))) void*)(lds + blk * 1024), 16, 0, 0);
        }
    }
    const float scale = p.sx[0] * p.sw[0];
    const unsigned call = rfl(cv) + 1;
    if (lane == 0) st_agent(cnt, call);
    unsigned* const tmo = (unsigned*)(myws + TMO_OFF);
    const unsigned* const flagp = (const unsigned*)(myws + FLAG_OFF) + (src * 16 + unit) * 32;
    const unsigned fbase = (call - 1) * NCHUNK;

    const bool local = (label == 0);
    const unsigned char* abase = local ? p.x : myws + (size_t)(call & 1) * STAGE_BYTES + (size_t)src * SHARD;
    const auto ar = __builtin_amdgcn_make_buffer_rsrc((void*)abase, 0, SHARD, 0x00020000);
    const unsigned cj = local ? 256u : CHUNK_BYTES, cs = local ? 128u : 4096u, ch = local ? 64u : 1024u;
    const unsigned va0 = local ? (lane & 15) * KDIM + (lane >> 4) * 16 : lane * 16;
    const unsigned va1 = va0 + (local ? 16u * KDIM : 2048u);
    const unsigned ubase = (unsigned)unit * UNIT_BYTES;

    __builtin_amdgcn_s_waitcnt(0x0F70);
    if (lane == 0) __hip_atomic_fetch_add((__attribute__((address_space(3))) unsigned*)(lds + LDS_W), 1u,
                                          __ATOMIC_RELAXED, __HIP_MEMORY_SCOPE_WORKGROUP);
    while (rfl(sync[0]) < 8) __builtin_amdgcn_s_sleep(2);
    if (!local && cw >= 4)
        while (rfl(sync[cw]) == 0) __builtin_amdgcn_s_sleep(8);

    unsigned avail = local ? 0xFFFFFFFFu : 0u;
    bool bad = false;
    auto wait_chunk = [&](int j) {
        const unsigned need = fbase + j + 1;
        if (avail >= need) return;
        const unsigned long long t0 = __builtin_amdgcn_s_memrealtime();
        unsigned spins = 0;
        while (true) {
            avail = rfl(ld_sys(flagp));
            if (avail >= need) break;
            if ((++spins & 63) == 0) {
                if (rfl(ld_agent(tmo)) == call || __builtin_amdgcn_s_memrealtime() - t0 > DEADLINE_TICKS) {
                    if (lane == 0) st_agent(tmo, call);
                    bad = true;
                    avail = 0xFFFFFFFFu;
                    break;
                }
            }
            __builtin_amdgcn_s_sleep(4);
        }
    };
    auto load_chunk = [&](u32x4 (&r)[8], int j) {
        const unsigned so = ubase + (unsigned)j * cj;
#pragma unroll
        for (int s = 0; s < 2; ++s)
#pragma unroll
            for (int ms = 0; ms < 2; ++ms)
#pragma unroll
                for (int h = 0; h < 2; ++h)
                    r[(s * 2 + ms) * 2 + h] =
                        __builtin_amdgcn_raw_buffer_load_b128(ar, ms ? va1 : va0, so + s * cs + h * ch, AUX_SYS);
        __builtin_amdgcn_sched_barrier(0);
    };
    f32x4 acc[2][2] = {};
    const int g = lane >> 4, li = lane & 15;
    const unsigned wb0 = g * 512 + (li >> 1) * 32 + (g & 1) * 16 + (li & 1) * 8;
    auto compute = [&](const u32x4 (&r)[8], int j) {
        const unsigned char* wp = lds + (unsigned)j * 8192;
#pragma unroll
        for (int s = 0; s < 2; ++s) {
            i32x8 wf[2];
#pragma unroll
            for (int t = 0; t < 2; ++t)
#pragma unroll
                for (int q = 0; q < 4; ++q) {
                    const i32x2 v = __builtin_amdgcn_ds_read_tr8_b64_v2i32(
                        (lds_i32x2_ptr)(wp + (wb0 ^ (t * 16)) + s * 4096 + (q >> 1) * 2048 + (q & 1) * 256));
                    wf[t][2 * q] = v[0];
                    wf[t][2 * q + 1] = v[1];
                }
#pragma unroll
            for (int ms = 0; ms < 2; ++ms) {
                const i32x8 xf = pack8(r[(s * 2 + ms) * 2], r[(s * 2 + ms) * 2 + 1]);
#pragma unroll
                for (int t = 0; t < 2; ++t)
                    acc[ms][t] = __builtin_amdgcn_mfma_scale_f32_16x16x128_f8f6f4(wf[t], xf, acc[ms][t], 1, 1, 0, 0, 0, 0);
            }
        }
    };

    u32x4 a0[8], a1[8];
    wait_chunk(0);
    load_chunk(a0, 0);
    for (int j = 1; j < NCHUNK - 1; j += 2) {
        wait_chunk(j);
        load_chunk(a1, j);
        compute(a0, j - 1);
        wait_chunk(j + 1);
        load_chunk(a0, j + 1);
        compute(a1, j);
    }
    wait_chunk(NCHUNK - 1);
    load_chunk(a1, NCHUNK - 1);
    const unsigned tv = ld_agent(tmo);
    if (cw < 4 && lane == 0) sync[cw + 4] = 1;
    compute(a0, NCHUNK - 2);
    compute(a1, NCHUNK - 1);

    const bool poison = bad || rfl(tv) == call;
    float* o = p.out + (size_t)(src * MLOC + unit * 32 + li) * NLOC + nblk * 32 + g * 4;
#pragma unroll
    for (int ms = 0; ms < 2; ++ms)
#pragma unroll
        for (int t = 0; t < 2; ++t) {
            f32x4 v;
#pragma unroll
            for (int e = 0; e < 4; ++e) {
                const float y = acc[ms][t][e] * scale;
                const float z = fminf(fmaxf(y, -60.f), 60.f);
                v[e] = poison ? __builtin_nanf("") : y * __builtin_amdgcn_rcpf(1.f + __expf(-z));
            }
            *(f32x4*)(o + (size_t)ms * 16 * NLOC + t * 16) = v;
        }
}

}

extern "C" void kernel_launch(int rank, int nranks, ncclComm_t comm, hipStream_t stream,
                              void* const* d_in, const int* in_sizes, int n_in,
                              void* d_out, int out_size,
                              const int* in_dtypes, int out_dtype,
                              void* const* d_out_all,
                              void* const* d_ws_all, size_t ws_bytes) {
    (void)comm; (void)in_dtypes; (void)out_dtype; (void)d_out_all;
    if (nranks != NR || n_in != 4 || (unsigned)rank >= (unsigned)NR || ws_bytes < WS_NEED ||
        in_sizes[0] != (int)SHARD || in_sizes[1] != KDIM * NLOC || out_size != NR * MLOC * NLOC) {
        fprintf(stderr, "unexpected problem shape\n");
        return;
    }
    Params p;
    p.x = (const unsigned char*)d_in[0];
    p.w = (const unsigned char*)d_in[1];
    p.sx = (const float*)d_in[2];
    p.sw = (const float*)d_in[3];
    p.out = (float*)d_out;
    for (int i = 0; i < NR; ++i) p.ws[i] = (unsigned char*)d_ws_all[i];
    p.rank = rank;
    hipLaunchKernelGGL(ag_gemm_silu_kernel, dim3(256), dim3(576), 0, stream, p);
    HIP_CHECK(hipGetLastError());
}
```

```cpp
#include <hip/hip_runtime.h>
#include <rccl/rccl.h>
#include <cstdio>
#include <cstdint>

#define HIP_CHECK(expr) do { hipError_t _e = (expr); if (_e != hipSuccess) { \
    fprintf(stderr, "HIP error %s at %s:%d\n", hipGetErrorString(_e), __FILE__, __LINE__); \
    return; } } while (0)

#define USE_AFFINITY 1
#define OUT_AUX 16

namespace {

constexpr int NR = 8;
constexpr int KDIM = 4096;
constexpr int MLOC = 512;
constexpr int NLOC = 512;
constexpr unsigned SHARD = MLOC * KDIM;
constexpr unsigned UNIT_BYTES = 32 * KDIM;
constexpr int NCHUNK = 16;
constexpr unsigned CHUNK_BYTES = 8192;
constexpr int NPUSH = 4;
constexpr int NBURST = 16 * NCHUNK / NPUSH;
constexpr size_t STAGE_BYTES = (size_t)NR * SHARD;
constexpr size_t FLAG_OFF = 2 * STAGE_BYTES;
constexpr size_t CNT_OFF = FLAG_OFF + 8 * 16 * 128;
constexpr size_t TMO_OFF = CNT_OFF + 256 * 64;
constexpr size_t WS_NEED = TMO_OFF + 128;
constexpr unsigned long long DEADLINE_TICKS = 40000000ull;
constexpr int LDS_W = KDIM * 32;
constexpr int AUX_SYS = 17;

typedef unsigned int u32x4 __attribute__((ext_vector_type(4)));
typedef int i32x2 __attribute__((ext_vector_type(2)));
typedef int i32x8 __attribute__((ext_vector_type(8)));
typedef float f32x4 __attribute__((ext_vector_type(4)));
typedef __attribute__((address_space(3))) i32x2* lds_i32x2_ptr;
typedef __attribute__((address_space(3))) volatile unsigned* lds_u32_ptr;

struct Params {
    const unsigned char* x;
    const unsigned char* w;
    const float* sx;
    const float* sw;
    float* out;
    unsigned char* ws[NR];
    int rank;
    unsigned peers;
};

__device__ __forceinline__ unsigned ld_sys(const unsigned* p) {
    return __hip_atomic_load(p, __ATOMIC_RELAXED, __HIP_MEMORY_SCOPE_SYSTEM);
}
__device__ __forceinline__ void st_sys(unsigned* p, unsigned v) {
    __hip_atomic_store(p, v, __ATOMIC_RELAXED, __HIP_MEMORY_SCOPE_SYSTEM);
}
__device__ __forceinline__ unsigned ld_agent(const unsigned* p) {
    return __hip_atomic_load(p, __ATOMIC_RELAXED, __HIP_MEMORY_SCOPE_AGENT);
}
__device__ __forceinline__ void st_agent(unsigned* p, unsigned v) {
    __hip_atomic_store(p, v, __ATOMIC_RELAXED, __HIP_MEMORY_SCOPE_AGENT);
}
__device__ __forceinline__ unsigned rfl(unsigned v) { return __builtin_amdgcn_readfirstlane(v); }

__device__ __forceinline__ i32x8 pack8(u32x4 lo, u32x4 hi) {
    i32x8 r;
    r[0] = lo[0]; r[1] = lo[1]; r[2] = lo[2]; r[3] = lo[3];
    r[4] = hi[0]; r[5] = hi[1]; r[6] = hi[2]; r[7] = hi[3];
    return r;
}

#define STAMP_DECL
#define STAMP_AT(i)
#define STAMP_OUT()

__global__ void __launch_bounds__(512)
ag_gemm_silu_kernel(Params p) {
    __shared__ __attribute__((aligned(16))) unsigned char lds[LDS_W + 64];
    const int lane = threadIdx.x & 63;
    const int wave = rfl(threadIdx.x >> 6);
    const int bid = blockIdx.x;
    const unsigned rank = (unsigned)p.rank & 7;
    unsigned char* const myws = p.ws[rank];
    unsigned* const cnt = (unsigned*)(myws + CNT_OFF) + bid * 16;
    const auto xr = __builtin_amdgcn_make_buffer_rsrc((void*)p.x, 0, SHARD, 0x00020000);
    const unsigned vx0 = (lane & 15) * KDIM + (lane >> 4) * 16;
    const unsigned vx1 = vx0 + 16 * KDIM;
    STAMP_DECL;
    STAMP_AT(0);

    int label = bid & 7, slot = bid >> 3;
    if (label != 0 && slot < NPUSH) {
        if (wave != 0) return;
        const unsigned cv = ld_agent(cnt);
        u32x4 d[2][8];
        auto load_burst = [&](u32x4 (&r)[8], int b) {
            b &= NBURST - 1;
            const unsigned so = (unsigned)(slot + NPUSH * (b >> 4)) * UNIT_BYTES + (unsigned)(b & 15) * 256;
#pragma unroll
            for (int s = 0; s < 2; ++s)
#pragma unroll
                for (int ms = 0; ms < 2; ++ms)
#pragma unroll
                    for (int h = 0; h < 2; ++h)
                        r[(s * 2 + ms) * 2 + h] =
                            __builtin_amdgcn_raw_buffer_load_b128(xr, (ms ? vx1 : vx0) + s * 128 + h * 64, so, 0);
        };
        load_burst(d[0], 0);
        load_burst(d[1], 1);
        const unsigned call = rfl(cv) + 1;
        if (lane == 0) st_agent(cnt, call);
        unsigned char* const dws = p.ws[(p.peers >> (4 * label)) & 7];
        const auto dr = __builtin_amdgcn_make_buffer_rsrc(
            dws + (size_t)(call & 1) * STAGE_BYTES + (size_t)rank * SHARD, 0, SHARD, 0x00020000);
        unsigned* const flags = (unsigned*)(dws + FLAG_OFF) + rank * 16 * 32;
        const unsigned fbase = (call - 1) * NCHUNK;
        const unsigned vs = lane * 16;
        auto store_burst = [&](const u32x4 (&r)[8], int b) {
            const unsigned so = (unsigned)(slot + NPUSH * (b >> 4)) * UNIT_BYTES + (unsigned)(b & 15) * CHUNK_BYTES;
#pragma unroll
            for (int i = 0; i < 8; ++i)
                __builtin_amdgcn_raw_buffer_store_b128(r[i], dr, vs + (i & 3) * 1024, so + (i >> 2) * 4096, AUX_SYS);
        };
        auto flag = [&](int b) {
            if (lane == 0) st_sys(flags + (slot + NPUSH * (b >> 4)) * 32, fbase + (b & 15) + 1);
            STAMP_AT(2 + b);
        };
        STAMP_AT(1);
        store_burst(d[0], 0);
        load_burst(d[0], 2);
        store_burst(d[1], 1);
        load_burst(d[1], 3);
        asm volatile("s_waitcnt vmcnt(24)" ::: "memory");
        flag(0);
        for (int b = 2; b < NBURST; b += 2) {
            store_burst(d[0], b);
            load_burst(d[0], b + 2);
            asm volatile("s_waitcnt vmcnt(25)" ::: "memory");
            flag(b - 1);
            store_burst(d[1], b + 1);
            load_burst(d[1], b + 3);
            asm volatile("s_waitcnt vmcnt(25)" ::: "memory");
            flag(b);
        }
        asm volatile("s_waitcnt vmcnt(0)" ::: "memory");
        flag(NBURST - 1);
        STAMP_OUT();
        return;
    }
    if (label == 0) {
        if (slot >= 7 * NPUSH) return;
        label = 1 + slot / NPUSH;
        slot = slot % NPUSH;
    }

    lds_u32_ptr sync = (lds_u32_ptr)(lds + LDS_W);
    const int nblk = slot & 15, rg = slot >> 4;
    const unsigned cv = ld_agent(cnt);
    if (wave == 0 && lane < 8) sync[lane] = 0;
    {
        const unsigned char* wsrc = p.w + (size_t)(lane >> 1) * NLOC + nblk * 32 + (((lane & 1) ^ ((lane >> 5) & 1)) * 16);
#pragma unroll
        for (int i = 0; i < 16; ++i) {
            const int blk = i * 8 + wave;
            __builtin_amdgcn_global_load_lds((const __attribute__((address_space(1))) void*)(wsrc + (size_t)blk * 32 * NLOC),
                                             (__attribute__((address_space(3))) void*)(lds + blk * 1024), 16, 0, 0);
        }
    }
    const float scale = p.sx[0] * p.sw[0];
    const unsigned call = rfl(cv) + 1;
    unsigned* const tmo = (unsigned*)(myws + TMO_OFF);
    const unsigned fbase = (call - 1) * NCHUNK;
    __builtin_amdgcn_s_waitcnt(0x0070);
    __builtin_amdgcn_s_barrier();
    if (threadIdx.x == 0) st_agent(cnt, call);
    STAMP_AT(1);

    const int g = lane >> 4, li = lane & 15;
    const unsigned wb0 = g * 512 + (li >> 1) * 32 + (g & 1) * 16 + (li & 1) * 8;
    f32x4 acc[2][2];
    auto compute = [&](const u32x4 (&r)[8], int j) {
        const unsigned char* wp = lds + (unsigned)j * 8192;
#pragma unroll
        for (int s = 0; s < 2; ++s) {
            i32x8 wf[2];
#pragma unroll
            for (int t = 0; t < 2; ++t)
#pragma unroll
                for (int q = 0; q < 4; ++q) {
                    const i32x2 v = __builtin_amdgcn_ds_read_tr8_b64_v2i32(
                        (lds_i32x2_ptr)(wp + (wb0 ^ (t * 16)) + s * 4096 + (q >> 1) * 2048 + (q & 1) * 256));
                    wf[t][2 * q] = v[0];
                    wf[t][2 * q + 1] = v[1];
                }
#pragma unroll
            for (int ms = 0; ms < 2; ++ms) {
                const i32x8 xf = pack8(r[(s * 2 + ms) * 2], r[(s * 2 + ms) * 2 + 1]);
#pragma unroll
                for (int t = 0; t < 2; ++t)
                    acc[ms][t] = __builtin_amdgcn_mfma_scale_f32_16x16x128_f8f6f4(wf[t], xf, acc[ms][t], 1, 1, 0, 0, 0, 0);
            }
        }
    };
    const auto orr = __builtin_amdgcn_make_buffer_rsrc(p.out, 0, NR * MLOC * NLOC * 4, 0x00020000);
    const float nsl2e = -scale * 1.44269504088896340736f;
    const float lim = 60.f * 1.44269504088896340736f;
    auto epilogue = [&](unsigned row0, bool poison) {
        const unsigned vo = ((row0 + li) * NLOC + nblk * 32 + g * 4) * 4;
        if (poison) {
#pragma unroll
            for (int i = 0; i < 4; ++i) acc[i >> 1][i & 1] = f32x4{__builtin_nanf(""), __builtin_nanf(""), __builtin_nanf(""), __builtin_nanf("")};
        }
#pragma unroll
        for (int ms = 0; ms < 2; ++ms)
#pragma unroll
            for (int t = 0; t < 2; ++t) {
                u32x4 v;
#pragma unroll
                for (int e = 0; e < 4; ++e) {
                    const float a = acc[ms][t][e];
                    const float ex = __builtin_amdgcn_exp2f(__builtin_amdgcn_fmed3f(a * nsl2e, -lim, lim));
                    v[e] = __builtin_bit_cast(unsigned, (a * scale) * __builtin_amdgcn_rcpf(1.f + ex));
                }
                __builtin_amdgcn_raw_buffer_store_b128(v, orr, vo + ms * 16 * NLOC * 4 + t * 64, 0, OUT_AUX);
            }
    };
    u32x4 a0[8], a1[8];

    {
        const int q = (label - 1) * 2 + rg;
        const int lu = wave == 7 ? q : (wave == 6 && q < 2 ? 14 + q : -1);
        if (lu >= 0) {
            auto load_local = [&](u32x4 (&r)[8], int j) {
                const unsigned so = (unsigned)lu * UNIT_BYTES + (unsigned)j * 256;
#pragma unroll
                for (int s = 0; s < 2; ++s)
#pragma unroll
                    for (int ms = 0; ms < 2; ++ms)
#pragma unroll
                        for (int h = 0; h < 2; ++h)
                            r[(s * 2 + ms) * 2 + h] =
                                __builtin_amdgcn_raw_buffer_load_b128(xr, (ms ? vx1 : vx0) + s * 128 + h * 64, so, 0);
                __builtin_amdgcn_sched_barrier(0);
            };
#pragma unroll
            for (int i = 0; i < 4; ++i) acc[i >> 1][i & 1] = f32x4{0.f, 0.f, 0.f, 0.f};
            load_local(a0, 0);
            for (int j = 0; j < NCHUNK; j += 2) {
                load_local(a1, j + 1);
                compute(a0, j);
                load_local(a0, (j + 2) & (NCHUNK - 1));
                compute(a1, j + 1);
            }
            epilogue(rank * MLOC + lu * 32, false);
            STAMP_AT(21);
        }
    }

    const int unit = 2 * wave + rg;
    const unsigned src = (p.peers >> (4 * label)) & 7;
    const unsigned* const flagp = (const unsigned*)(myws + FLAG_OFF) + (src * 16 + unit) * 32;
    const auto ar = __builtin_amdgcn_make_buffer_rsrc(
        myws + (size_t)(call & 1) * STAGE_BYTES + (size_t)src * SHARD, 0, SHARD, 0x00020000);
    const unsigned va = lane * 16;
    const unsigned ubase = (unsigned)unit * UNIT_BYTES;
    if (wave >= NPUSH / 2)
        while (rfl(sync[wave]) == 0) __builtin_amdgcn_s_sleep(8);

    unsigned avail = 0;
    bool bad = false;
    auto wait_chunk = [&](int j) {
        const unsigned need = fbase + j + 1;
        if (avail >= need) return;
        const unsigned long long t0 = __builtin_amdgcn_s_memrealtime();
        unsigned spins = 0;
        while (true) {
            avail = rfl(ld_sys(flagp));
            if (avail >= need) break;
            if ((++spins & 63) == 0) {
                if (rfl(ld_agent(tmo)) == call || __builtin_amdgcn_s_memrealtime() - t0 > DEADLINE_TICKS) {
                    if (lane == 0) st_agent(tmo, call);
                    bad = true;
                    avail = 0xFFFFFFFFu;
                    break;
                }
            }
            __builtin_amdgcn_s_sleep(2);
        }
    };
    auto load_chunk = [&](u32x4 (&r)[8], int j) {
        const unsigned so = ubase + (unsigned)j * CHUNK_BYTES;
        STAMP_AT(2 + j);
#pragma unroll
        for (int i = 0; i < 8; ++i)
            r[i] = __builtin_amdgcn_raw_buffer_load_b128(ar, va + (i & 3) * 1024, so + (i >> 2) * 4096, AUX_SYS);
        __builtin_amdgcn_sched_barrier(0);
    };
#pragma unroll
    for (int i = 0; i < 4; ++i) acc[i >> 1][i & 1] = f32x4{0.f, 0.f, 0.f, 0.f};
    wait_chunk(0);
    load_chunk(a0, 0);
    for (int j = 1; j < NCHUNK - 1; j += 2) {
        wait_chunk(j);
        load_chunk(a1, j);
        compute(a0, j - 1);
        wait_chunk(j + 1);
        load_chunk(a0, j + 1);
        compute(a1, j);
    }
    wait_chunk(NCHUNK - 1);
    load_chunk(a1, NCHUNK - 1);
    const unsigned tv = ld_agent(tmo);
    if (wave + NPUSH / 2 < 8 && lane == 0) sync[wave + NPUSH / 2] = 1;
    compute(a0, NCHUNK - 2);
    compute(a1, NCHUNK - 1);
    STAMP_AT(18);
    epilogue(src * MLOC + unit * 32, bad || rfl(tv) == call);
}

}

extern "C" void kernel_launch(int rank, int nranks, ncclComm_t comm, hipStream_t stream,
                              void* const* d_in, const int* in_sizes, int n_in,
                              void* d_out, int out_size,
                              const int* in_dtypes, int out_dtype,
                              void* const* d_out_all,
                              void* const* d_ws_all, size_t ws_bytes) {
    (void)comm; (void)in_dtypes; (void)out_dtype; (void)d_out_all;
    if (nranks != NR || n_in != 4 || (unsigned)rank >= (unsigned)NR || ws_bytes < WS_NEED ||
        in_sizes[0] != (int)SHARD || in_sizes[1] != KDIM * NLOC || out_size != NR * MLOC * NLOC) {
        fprintf(stderr, "unexpected problem shape\n");
        return;
    }
    Params p;
    p.x = (const unsigned char*)d_in[0];
    p.w = (const unsigned char*)d_in[1];
    p.sx = (const float*)d_in[2];
    p.sw = (const float*)d_in[3];
    p.out = (float*)d_out;
    for (int i = 0; i < NR; ++i) p.ws[i] = (unsigned char*)d_ws_all[i];
    p.rank = rank;
    {
        int dev = -1;
        const bool ident = hipGetDevice(&dev) == hipSuccess && dev == rank;
        const unsigned grp = 0xB2u;
        unsigned t = (unsigned)rank;
        int l = 1;
        if (ident && USE_AFFINITY) {
            for (int pass = 0; pass < 2; ++pass)
                for (int i = 1; i < NR; ++i) {
                    const int q = (rank + i) & 7;
                    const bool same = ((grp >> q) & 1) == ((grp >> rank) & 1);
                    if (same == (pass == 0)) t |= (unsigned)q << (4 * l++);
                }
        } else {
            for (int i = 1; i < NR; ++i) t |= (unsigned)((rank + i) & 7) << (4 * l++);
        }
        p.peers = t;
    }
    hipLaunchKernelGGL(ag_gemm_silu_kernel, dim3(256), dim3(512), 0, stream, p);
    HIP_CHECK(hipGetLastError());
}
```

```cpp
#include <hip/hip_runtime.h>
#include <rccl/rccl.h>
#include <cstdio>
#include <cstdint>

#define HIP_CHECK(expr) do { hipError_t _e = (expr); if (_e != hipSuccess) { \
    fprintf(stderr, "HIP error %s at %s:%d\n", hipGetErrorString(_e), __FILE__, __LINE__); \
    return; } } while (0)

#define USE_AFFINITY 1
#define USE_LL 1
#define UNIT_SKEW 4096
#define PEEL 0
#define FLAG_ALL 1
#define POLL_SLEEP 2
#define W_DELAY 0
#define HELPER_DELAY 0
#define OUT_AUX 16

namespace {

constexpr int NR = 8;
constexpr int KDIM = 4096;
constexpr int MLOC = 512;
constexpr int NLOC = 512;
constexpr unsigned SHARD = MLOC * KDIM;
constexpr unsigned UNIT_BYTES = 32 * KDIM;
constexpr int NCHUNK = 16;
constexpr unsigned CHUNK_BYTES = 8192;
constexpr int NPUSH = 4;
constexpr int NBURST = 16 * NCHUNK / NPUSH;
constexpr int NTAIL = 4;
constexpr unsigned USTRIDE = UNIT_BYTES + UNIT_SKEW;
constexpr unsigned SLOT = 16 * USTRIDE;
constexpr size_t STAGE_BYTES = (size_t)NR * SLOT;
constexpr size_t FLAG_OFF = 2 * STAGE_BYTES;
constexpr size_t CNT_OFF = FLAG_OFF + 8 * 16 * 128;
constexpr size_t TMO_OFF = CNT_OFF + 256 * 64;
constexpr size_t WS_NEED = TMO_OFF + 128;
constexpr unsigned DEADLINE_TICKS = 40000000u;
constexpr int LDS_W = KDIM * 32;
constexpr int AUX_SYS = 17;

typedef unsigned int u32x4 __attribute__((ext_vector_type(4)));
typedef int i32x2 __attribute__((ext_vector_type(2)));
typedef int i32x8 __attribute__((ext_vector_type(8)));
typedef float f32x4 __attribute__((ext_vector_type(4)));
typedef __attribute__((address_space(3))) i32x2* lds_i32x2_ptr;
typedef __attribute__((address_space(3))) volatile unsigned* lds_u32_ptr;

struct Params {
    const unsigned char* x;
    const unsigned char* w;
    const float* sx;
    const float* sw;
    float* out;
    unsigned char* ws[NR];
    int rank;
    unsigned peers;
};

__device__ __forceinline__ unsigned ld_sys(const unsigned* p) {
    return __hip_atomic_load(p, __ATOMIC_RELAXED, __HIP_MEMORY_SCOPE_SYSTEM);
}
__device__ __forceinline__ void st_sys(unsigned* p, unsigned v) {
    __hip_atomic_store(p, v, __ATOMIC_RELAXED, __HIP_MEMORY_SCOPE_SYSTEM);
}
__device__ __forceinline__ unsigned ld_agent(const unsigned* p) {
    return __hip_atomic_load(p, __ATOMIC_RELAXED, __HIP_MEMORY_SCOPE_AGENT);
}
__device__ __forceinline__ void st_agent(unsigned* p, unsigned v) {
    __hip_atomic_store(p, v, __ATOMIC_RELAXED, __HIP_MEMORY_SCOPE_AGENT);
}
__device__ __forceinline__ unsigned rfl(unsigned v) { return __builtin_amdgcn_readfirstlane(v); }

__device__ __forceinline__ i32x8 pack8(u32x4 lo, u32x4 hi) {
    i32x8 r;
    r[0] = lo[0]; r[1] = lo[1]; r[2] = lo[2]; r[3] = lo[3];
    r[4] = hi[0]; r[5] = hi[1]; r[6] = hi[2]; r[7] = hi[3];
    return r;
}

#define STAMP_DECL
#define STAMP_AT(i)
#define STAMP_OUT()

__global__ void __launch_bounds__(512)
ag_gemm_silu_kernel(Params p) {
    __shared__ __attribute__((aligned(16))) unsigned char lds[LDS_W + 64];
    const int lane = threadIdx.x & 63;
    const int wave = rfl(threadIdx.x >> 6);
    const int bid = blockIdx.x;
    const unsigned rank = (unsigned)p.rank & 7;
    unsigned char* const myws = p.ws[rank];
    unsigned* const cnt = (unsigned*)(myws + CNT_OFF) + bid * 16;
    const auto xr = __builtin_amdgcn_make_buffer_rsrc((void*)p.x, 0, SHARD, 0x00020000);
    const unsigned vx0 = (lane & 15) * KDIM + (lane >> 4) * 16;
    const unsigned vx1 = vx0 + 16 * KDIM;
    STAMP_DECL;
    STAMP_AT(0);

    int label = bid & 7, slot = bid >> 3;
    if (label != 0 && slot < NPUSH) {
        if (wave > 1) return;
        const unsigned cv = ld_agent(cnt);
        if (wave == 1) {
            const unsigned call = rfl(cv) + 1;
            const auto zr = __builtin_amdgcn_make_buffer_rsrc(
                myws + (size_t)((call + 1) & 1) * STAGE_BYTES + (size_t)((p.peers >> (4 * label)) & 7) * SLOT, 0, SLOT, 0x00020000);
            const unsigned zo = (unsigned)(16 - NPUSH + slot) * USTRIDE + (NCHUNK - NTAIL) * CHUNK_BYTES + lane * 16;
            if (HELPER_DELAY) __builtin_amdgcn_s_sleep(HELPER_DELAY);
#pragma unroll
            for (int i = 0; i < NTAIL * 8; ++i)
                __builtin_amdgcn_raw_buffer_store_b128(u32x4{0u, 0u, 0u, 0u}, zr, zo + i * 1024, 0, AUX_SYS);
            asm volatile("s_waitcnt vmcnt(0)" ::: "memory");
            __builtin_amdgcn_s_barrier();
            return;
        }
        u32x4 d[2][8];
        auto load_burst = [&](u32x4 (&r)[8], int b) {
            b &= NBURST - 1;
            const unsigned so = (unsigned)(slot + NPUSH * (b >> 4)) * UNIT_BYTES + (unsigned)(b & 15) * 256;
#pragma unroll
            for (int s = 0; s < 2; ++s)
#pragma unroll
                for (int ms = 0; ms < 2; ++ms)
#pragma unroll
                    for (int h = 0; h < 2; ++h)
                        r[(s * 2 + ms) * 2 + h] =
                            __builtin_amdgcn_raw_buffer_load_b128(xr, (ms ? vx1 : vx0) + s * 128 + h * 64, so, 0);
        };
        load_burst(d[0], 0);
        load_burst(d[1], 1);
        const unsigned call = rfl(cv) + 1;
        unsigned char* const dws = p.ws[(p.peers >> (4 * label)) & 7];
        const auto dr = __builtin_amdgcn_make_buffer_rsrc(
            dws + (size_t)(call & 1) * STAGE_BYTES + (size_t)rank * SLOT, 0, SLOT, 0x00020000);
        unsigned* const flags = (unsigned*)(dws + FLAG_OFF) + rank * 16 * 32;
        const unsigned fbase = (call - 1) * NCHUNK;
        const unsigned vs = lane * 16;
        auto store_burst = [&](const u32x4 (&r)[8], int b) {
            const unsigned so = (unsigned)(slot + NPUSH * (b >> 4)) * USTRIDE + (unsigned)(b & 15) * CHUNK_BYTES;
#pragma unroll
            for (int i = 0; i < 8; ++i)
                __builtin_amdgcn_raw_buffer_store_b128(r[i], dr, vs + (i & 3) * 1024, so + (i >> 2) * 4096, AUX_SYS);
        };
        auto flag = [&](int b) {
            if (FLAG_ALL || lane == 0) st_sys(flags + (slot + NPUSH * (b >> 4)) * 32, fbase + (b & 15) + 1);
            STAMP_AT(2 + b);
        };
        STAMP_AT(1);
        store_burst(d[0], 0);
        load_burst(d[0], 2);
        store_burst(d[1], 1);
        load_burst(d[1], 3);
        asm volatile("s_waitcnt vmcnt(24)" ::: "memory");
        flag(0);
        for (int b = 2; b < NBURST - (PEEL ? 2 : 0); b += 2) {
            if (b == NBURST / 2) {
                __builtin_amdgcn_s_barrier();
                st_agent(cnt, call);
            }
            store_burst(d[0], b);
            load_burst(d[0], b + 2);
            asm volatile("s_waitcnt vmcnt(25)" ::: "memory");
            flag(b - 1);
            store_burst(d[1], b + 1);
            load_burst(d[1], b + 3);
            asm volatile("s_waitcnt vmcnt(25)" ::: "memory");
            flag(b);
        }
        if (PEEL) {
            store_burst(d[0], NBURST - 2);
            asm volatile("s_waitcnt vmcnt(17)" ::: "memory");
            flag(NBURST - 3);
            store_burst(d[1], NBURST - 1);
            asm volatile("s_waitcnt vmcnt(9)" ::: "memory");
            flag(NBURST - 2);
            asm volatile("s_waitcnt vmcnt(1)" ::: "memory");
        } else {
            asm volatile("s_waitcnt vmcnt(9)" ::: "memory");
        }
        flag(NBURST - 1);
        STAMP_OUT();
        return;
    }
    if (label == 0) {
        if (slot >= 7 * NPUSH) return;
        label = 1 + slot / NPUSH;
        slot = slot % NPUSH;
    }

    lds_u32_ptr sync = (lds_u32_ptr)(lds + LDS_W);
    const int nblk = slot & 15, rg = slot >> 4;
    const unsigned cv = ld_agent(cnt);
    if (wave == 0 && lane < 8) sync[lane] = 0;
    if (W_DELAY) __builtin_amdgcn_s_sleep(W_DELAY);
    {
        const unsigned char* wsrc = p.w + (size_t)(lane >> 1) * NLOC + nblk * 32 + (((lane & 1) ^ ((lane >> 5) & 1)) * 16);
#pragma unroll
        for (int i = 0; i < 16; ++i) {
            const int blk = i * 8 + wave;
            __builtin_amdgcn_global_load_lds((const __attribute__((address_space(1))) void*)(wsrc + (size_t)blk * 32 * NLOC),
                                             (__attribute__((address_space(3))) void*)(lds + blk * 1024), 16, 0, 0);
        }
    }
    const float scale = p.sx[0] * p.sw[0];
    const unsigned call = rfl(cv) + 1;
    unsigned* const tmo = (unsigned*)(myws + TMO_OFF);
    const unsigned fbase = (call - 1) * NCHUNK;
    __builtin_amdgcn_s_waitcnt(0x0070);
    __builtin_amdgcn_s_barrier();
    if (threadIdx.x == 0) st_agent(cnt, call);
    STAMP_AT(1);

    const int g = lane >> 4, li = lane & 15;
    const unsigned wb0 = g * 512 + (li >> 1) * 32 + (g & 1) * 16 + (li & 1) * 8;
    f32x4 acc[2][2];
    i32x8 wfa[2], wfb[2];
    auto read_w = [&](i32x8 (&wf)[2], int n) {
        const unsigned char* wp = lds + (unsigned)n * 4096;
#pragma unroll
        for (int t = 0; t < 2; ++t)
#pragma unroll
            for (int q = 0; q < 4; ++q) {
                const i32x2 v = __builtin_amdgcn_ds_read_tr8_b64_v2i32(
                    (lds_i32x2_ptr)(wp + (wb0 ^ (t * 16)) + (q >> 1) * 2048 + (q & 1) * 256));
                wf[t][2 * q] = v[0];
                wf[t][2 * q + 1] = v[1];
            }
        __builtin_amdgcn_sched_barrier(0);
    };
    auto mma = [&](const i32x8 (&wf)[2], const u32x4 (&r)[8], int s) {
#pragma unroll
        for (int ms = 0; ms < 2; ++ms) {
            const i32x8 xf = pack8(r[(s * 2 + ms) * 2], r[(s * 2 + ms) * 2 + 1]);
#pragma unroll
            for (int t = 0; t < 2; ++t)
                acc[ms][t] = __builtin_amdgcn_mfma_scale_f32_16x16x128_f8f6f4(wf[t], xf, acc[ms][t], 1, 1, 0, 0, 0, 0);
        }
        __builtin_amdgcn_sched_barrier(0);
    };
    auto compute = [&](const u32x4 (&r)[8], int j) {
        read_w(wfb, 2 * j + 1);
        mma(wfa, r, 0);
        read_w(wfa, (2 * j + 2) & (2 * NCHUNK - 1));
        mma(wfb, r, 1);
    };
    read_w(wfa, 0);
    const auto orr = __builtin_amdgcn_make_buffer_rsrc(p.out, 0, NR * MLOC * NLOC * 4, 0x00020000);
    const float nsl2e = -scale * 1.44269504088896340736f;
    const float lim = 60.f * 1.44269504088896340736f;
    auto epilogue = [&](unsigned row0, bool poison) {
        const unsigned vo = ((row0 + li) * NLOC + nblk * 32 + g * 4) * 4;
        if (poison) {
#pragma unroll
            for (int i = 0; i < 4; ++i) acc[i >> 1][i & 1] = f32x4{__builtin_nanf(""), __builtin_nanf(""), __builtin_nanf(""), __builtin_nanf("")};
        }
#pragma unroll
        for (int ms = 0; ms < 2; ++ms)
#pragma unroll
            for (int t = 0; t < 2; ++t) {
                u32x4 v;
#pragma unroll
                for (int e = 0; e < 4; ++e) {
                    const float a = acc[ms][t][e];
                    const float ex = __builtin_amdgcn_exp2f(__builtin_amdgcn_fmed3f(a * nsl2e, -lim, lim));
                    v[e] = __builtin_bit_cast(unsigned, (a * scale) * __builtin_amdgcn_rcpf(1.f + ex));
                }
                __builtin_amdgcn_raw_buffer_store_b128(v, orr, vo + ms * 16 * NLOC * 4 + t * 64, 0, OUT_AUX);
            }
    };
    u32x4 a0[8], a1[8];

    {
        const int q = (label - 1) * 2 + rg;
        const int lu = wave == 7 ? q : (wave == 6 && q < 2 ? 14 + q : -1);
        if (lu >= 0) {
            auto load_local = [&](u32x4 (&r)[8], int j) {
                const unsigned so = (unsigned)lu * UNIT_BYTES + (unsigned)j * 256;
#pragma unroll
                for (int s = 0; s < 2; ++s)
#pragma unroll
                    for (int ms = 0; ms < 2; ++ms)
#pragma unroll
                        for (int h = 0; h < 2; ++h)
                            r[(s * 2 + ms) * 2 + h] =
                                __builtin_amdgcn_raw_buffer_load_b128(xr, (ms ? vx1 : vx0) + s * 128 + h * 64, so, 0);
                __builtin_amdgcn_sched_barrier(0);
            };
#pragma unroll
            for (int i = 0; i < 4; ++i) acc[i >> 1][i & 1] = f32x4{0.f, 0.f, 0.f, 0.f};
            load_local(a0, 0);
            for (int j = 0; j < NCHUNK; j += 2) {
                load_local(a1, j + 1);
                compute(a0, j);
                load_local(a0, (j + 2) & (NCHUNK - 1));
                compute(a1, j + 1);
            }
            epilogue(rank * MLOC + lu * 32, false);
            STAMP_AT(21);
        }
    }

    const int unit = 2 * wave + rg;
    const unsigned src = (p.peers >> (4 * label)) & 7;
    const unsigned* const flagp = (const unsigned*)(myws + FLAG_OFF) + (src * 16 + unit) * 32;
    const auto ar = __builtin_amdgcn_make_buffer_rsrc(
        myws + (size_t)(call & 1) * STAGE_BYTES + (size_t)src * SLOT, 0, SLOT, 0x00020000);
    const unsigned va = lane * 16;
    const unsigned ubase = (unsigned)unit * USTRIDE;
    if (wave >= NPUSH / 2)
        while (rfl(sync[wave]) == 0) __builtin_amdgcn_s_sleep(8);

    unsigned avail = 0;
    bool bad = false;
    unsigned spins = 0;
    unsigned twait = 0;
    auto expired = [&]() -> bool {
        if ((++spins & 63) != 0) return false;
        const unsigned now = rfl((unsigned)__builtin_amdgcn_s_memrealtime());
        __builtin_amdgcn_s_waitcnt(0xC07F);
        if (spins == 64) twait = now;
        if (rfl(ld_agent(tmo)) != call && now - twait <= DEADLINE_TICKS) return false;
        st_agent(tmo, call);
        bad = true;
        return true;
    };
    auto wait_chunk = [&](int j) {
        const unsigned need = fbase + j + 1;
        while (avail < need) {
            avail = rfl(ld_sys(flagp));
            if (avail >= need) break;
            if (expired()) avail = 0xFFFFFFFFu;
            __builtin_amdgcn_s_sleep(POLL_SLEEP);
        }
    };
    auto load_chunk = [&](u32x4 (&r)[8], int j) {
        const unsigned so = ubase + (unsigned)j * CHUNK_BYTES;
        STAMP_AT(2 + j);
#pragma unroll
        for (int i = 0; i < 8; ++i)
            r[i] = __builtin_amdgcn_raw_buffer_load_b128(ar, va + (i & 3) * 1024, so + (i >> 2) * 4096, AUX_SYS);
        __builtin_amdgcn_sched_barrier(0);
    };
#pragma unroll
    for (int i = 0; i < 4; ++i) acc[i >> 1][i & 1] = f32x4{0.f, 0.f, 0.f, 0.f};
    wait_chunk(0);
    load_chunk(a0, 0);
    if (!USE_LL || wave < 8 - NPUSH / 2) {
        for (int j = 1; j < NCHUNK - 1; j += 2) {
            wait_chunk(j);
            load_chunk(a1, j);
            compute(a0, j - 1);
            wait_chunk(j + 1);
            load_chunk(a0, j + 1);
            compute(a1, j);
        }
        wait_chunk(NCHUNK - 1);
        load_chunk(a1, NCHUNK - 1);
        const unsigned tv = ld_agent(tmo);
        if (wave + NPUSH / 2 < 8) sync[wave + NPUSH / 2] = 1;
        compute(a0, NCHUNK - 2);
        compute(a1, NCHUNK - 1);
        bad = bad || rfl(tv) == call;
    } else {
        for (int j = 1; j < NCHUNK - NTAIL - 1; j += 2) {
            wait_chunk(j);
            load_chunk(a1, j);
            compute(a0, j - 1);
            wait_chunk(j + 1);
            load_chunk(a0, j + 1);
            compute(a1, j);
        }
        constexpr int T0 = NCHUNK - NTAIL;
        wait_chunk(T0 - 1);
        load_chunk(a1, T0 - 1);
        static_assert(NTAIL == 4, "tail code is written for 4 chunks");
        u32x4 t0[8], t1[8];
        unsigned cover = avail;
        load_chunk(t0, T0);
        load_chunk(t1, T0 + 1);
        const unsigned tv = ld_agent(tmo);
        compute(a0, T0 - 2);
        load_chunk(a0, T0 + 2);
        compute(a1, T0 - 1);
        load_chunk(a1, T0 + 3);
        bad = bad || rfl(tv) == call;
        auto settle = [&](u32x4 (&r)[8], int c, auto&& reload) {
            const unsigned need = fbase + T0 + c + 1;
            while (cover < need) {
                unsigned m = 0xFFFFFFFFu;
#pragma unroll
                for (int i = 0; i < 8; ++i) m = min(min(m, min(r[i][0], r[i][1])), min(r[i][2], r[i][3]));
                if (__builtin_amdgcn_ballot_w64(m == 0) == 0) break;
                cover = avail;
                const unsigned fv = ld_sys(flagp);
                reload();
                avail = rfl(fv);
                if (expired()) cover = avail = 0xFFFFFFFFu;
            }
            STAMP_AT(24 + c);
            compute(r, T0 + c);
        };
        settle(t0, 0, [&]() { load_chunk(t0, T0); load_chunk(t1, T0 + 1); load_chunk(a0, T0 + 2); load_chunk(a1, T0 + 3); });
        settle(t1, 1, [&]() { load_chunk(t1, T0 + 1); load_chunk(a0, T0 + 2); load_chunk(a1, T0 + 3); });
        settle(a0, 2, [&]() { load_chunk(a0, T0 + 2); load_chunk(a1, T0 + 3); });
        settle(a1, 3, [&]() { load_chunk(a1, T0 + 3); });
    }
    STAMP_AT(18);
    epilogue(src * MLOC + unit * 32, bad);
}

}

extern "C" void kernel_launch(int rank, int nranks, ncclComm_t comm, hipStream_t stream,
                              void* const* d_in, const int* in_sizes, int n_in,
                              void* d_out, int out_size,
                              const int* in_dtypes, int out_dtype,
                              void* const* d_out_all,
                              void* const* d_ws_all, size_t ws_bytes) {
    (void)comm; (void)in_dtypes; (void)out_dtype; (void)d_out_all;
    if (nranks != NR || n_in != 4 || (unsigned)rank >= (unsigned)NR || ws_bytes < WS_NEED ||
        in_sizes[0] != (int)SHARD || in_sizes[1] != KDIM * NLOC || out_size != NR * MLOC * NLOC) {
        fprintf(stderr, "unexpected problem shape\n");
        return;
    }
    Params p;
    p.x = (const unsigned char*)d_in[0];
    p.w = (const unsigned char*)d_in[1];
    p.sx = (const float*)d_in[2];
    p.sw = (const float*)d_in[3];
    p.out = (float*)d_out;
    for (int i = 0; i < NR; ++i) p.ws[i] = (unsigned char*)d_ws_all[i];
    p.rank = rank;
    {
        int dev = -1;
        const bool ident = hipGetDevice(&dev) == hipSuccess && dev == rank;
        const unsigned grp = 0xB2u;
        unsigned t = (unsigned)rank;
        int l = 1;
        if (ident && USE_AFFINITY) {
            for (int pass = 0; pass < 2; ++pass)
                for (int i = 1; i < NR; ++i) {
                    const int q = (rank + i) & 7;
                    const bool same = ((grp >> q) & 1) == ((grp >> rank) & 1);
                    if (same == (pass == 0)) t |= (unsigned)q << (4 * l++);
                }
        } else {
            for (int i = 1; i < NR; ++i) t |= (unsigned)((rank + i) & 7) << (4 * l++);
        }
        p.peers = t;
    }
    hipLaunchKernelGGL(ag_gemm_silu_kernel, dim3(256), dim3(512), 0, stream, p);
    HIP_CHECK(hipGetLastError());
}
```

```cpp
#include <hip/hip_runtime.h>
#include <rccl/rccl.h>
#include <cstdio>
#include <cstdint>

#define HIP_CHECK(expr) do { hipError_t _e = (expr); if (_e != hipSuccess) { \
    fprintf(stderr, "HIP error %s at %s:%d\n", hipGetErrorString(_e), __FILE__, __LINE__); \
    return; } } while (0)

namespace {

constexpr int NR = 8;
constexpr int KDIM = 4096;
constexpr int MLOC = 512;
constexpr int NLOC = 512;
constexpr unsigned SHARD = MLOC * KDIM;
constexpr unsigned UNIT_BYTES = 32 * KDIM;
constexpr int NCHUNK = 16;
constexpr unsigned CHUNK_BYTES = 8192;
constexpr int NPUSH = 4;
constexpr int NBURST = 16 * NCHUNK / NPUSH;
constexpr int NTAIL = 4;
constexpr unsigned long long FLAG_MASK = 0x8faa800080008000ull;
constexpr unsigned USTRIDE = UNIT_BYTES + 4096;
constexpr unsigned SLOT = 16 * USTRIDE;
constexpr size_t STAGE_BYTES = (size_t)NR * SLOT;
constexpr size_t FLAG_OFF = 2 * STAGE_BYTES;
constexpr size_t CNT_OFF = FLAG_OFF + 8 * 16 * 128;
constexpr size_t TMO_OFF = CNT_OFF + 256 * 64;
constexpr size_t WS_NEED = TMO_OFF + 128;
constexpr unsigned DEADLINE_TICKS = 40000000u;
constexpr int LDS_W = KDIM * 32;
constexpr int AUX_SYS = 17;
constexpr int AUX_WT = 16;
constexpr unsigned RSRC3 = 0x00020000;

typedef unsigned int u32x4 __attribute__((ext_vector_type(4)));
typedef int i32x2 __attribute__((ext_vector_type(2)));
typedef int i32x8 __attribute__((ext_vector_type(8)));
typedef float f32x4 __attribute__((ext_vector_type(4)));
typedef __attribute__((address_space(3))) i32x2* lds_i32x2_ptr;
typedef __attribute__((address_space(3))) volatile unsigned* lds_u32_ptr;

struct Params {
    const unsigned char* x;
    const unsigned char* w;
    const float* sx;
    const float* sw;
    float* out;
    unsigned char* ws[NR];
    int rank;
    unsigned peers;
};

__device__ __forceinline__ unsigned ld_sys(const unsigned* p) {
    return __hip_atomic_load(p, __ATOMIC_RELAXED, __HIP_MEMORY_SCOPE_SYSTEM);
}
__device__ __forceinline__ void st_sys(unsigned* p, unsigned v) {
    __hip_atomic_store(p, v, __ATOMIC_RELAXED, __HIP_MEMORY_SCOPE_SYSTEM);
}
__device__ __forceinline__ unsigned ld_agent(const unsigned* p) {
    return __hip_atomic_load(p, __ATOMIC_RELAXED, __HIP_MEMORY_SCOPE_AGENT);
}
__device__ __forceinline__ void st_agent(unsigned* p, unsigned v) {
    __hip_atomic_store(p, v, __ATOMIC_RELAXED, __HIP_MEMORY_SCOPE_AGENT);
}
__device__ __forceinline__ unsigned rfl(unsigned v) { return __builtin_amdgcn_readfirstlane(v); }
__device__ __forceinline__ void compiler_fence() { asm volatile("" ::: "memory"); }

__device__ __forceinline__ i32x8 pack8(u32x4 lo, u32x4 hi) {
    i32x8 r;
    r[0] = lo[0]; r[1] = lo[1]; r[2] = lo[2]; r[3] = lo[3];
    r[4] = hi[0]; r[5] = hi[1]; r[6] = hi[2]; r[7] = hi[3];
    return r;
}

__global__ void __launch_bounds__(512)
ag_gemm_silu_kernel(Params p) {
    __shared__ __attribute__((aligned(16))) unsigned char lds[LDS_W + 64];
    const int lane = threadIdx.x & 63;
    const int wave = rfl(threadIdx.x >> 6);
    const int bid = blockIdx.x;
    const unsigned rank = (unsigned)p.rank & 7;
    unsigned char* const myws = p.ws[rank];
    unsigned* const cnt = (unsigned*)(myws + CNT_OFF) + bid * 16;
    const auto xr = __builtin_amdgcn_make_buffer_rsrc((void*)p.x, 0, SHARD, RSRC3);
    const unsigned vx0 = (lane & 15) * KDIM + (lane >> 4) * 16;
    const unsigned vx1 = vx0 + 16 * KDIM;
    auto load_x = [&](u32x4 (&r)[8], unsigned so) {
#pragma unroll
        for (int s = 0; s < 2; ++s)
#pragma unroll
            for (int ms = 0; ms < 2; ++ms)
#pragma unroll
                for (int h = 0; h < 2; ++h)
                    r[(s * 2 + ms) * 2 + h] =
                        __builtin_amdgcn_raw_buffer_load_b128(xr, (ms ? vx1 : vx0) + s * 128 + h * 64, so, 0);
    };

    int label = bid & 7, slot = bid >> 3;
    if (label != 0 && slot < NPUSH) {
        if (wave > 1) return;
        const unsigned peer = (p.peers >> (4 * label)) & 7;
        const unsigned cv = ld_agent(cnt);
        if (wave == 1) {
            const unsigned call = rfl(cv) + 1;
            const auto zr = __builtin_amdgcn_make_buffer_rsrc(
                myws + (size_t)((call + 1) & 1) * STAGE_BYTES + (size_t)peer * SLOT, 0, SLOT, RSRC3);
            const unsigned zo = (unsigned)(16 - NPUSH + slot) * USTRIDE + (NCHUNK - NTAIL) * CHUNK_BYTES + lane * 16;
#pragma unroll
            for (int i = 0; i < NTAIL * 8; ++i)
                __builtin_amdgcn_raw_buffer_store_b128(u32x4{0u, 0u, 0u, 0u}, zr, zo + i * 1024, 0, AUX_SYS);
            asm volatile("s_waitcnt vmcnt(0)" ::: "memory");
            __builtin_amdgcn_s_barrier();
            return;
        }
        u32x4 d[2][8];
        auto load_burst = [&](u32x4 (&r)[8], int b) {
            b &= NBURST - 1;
            load_x(r, (unsigned)(slot + NPUSH * (b >> 4)) * UNIT_BYTES + (unsigned)(b & 15) * 256);
        };
        load_burst(d[0], 0);
        load_burst(d[1], 1);
        const unsigned call = rfl(cv) + 1;
        unsigned char* const dws = p.ws[peer];
        const auto dr = __builtin_amdgcn_make_buffer_rsrc(
            dws + (size_t)(call & 1) * STAGE_BYTES + (size_t)rank * SLOT, 0, SLOT, RSRC3);
        unsigned* const flags = (unsigned*)(dws + FLAG_OFF) + rank * 16 * 32;
        const unsigned fbase = (call - 1) * NCHUNK;
        unsigned* const scratch = cnt + 8;
        const long long fdelta = (char*)flags - (char*)scratch;
        const unsigned vs = lane * 16;
        auto store_burst = [&](const u32x4 (&r)[8], int b) {
            const unsigned so = (unsigned)(slot + NPUSH * (b >> 4)) * USTRIDE + (unsigned)(b & 15) * CHUNK_BYTES;
#pragma unroll
            for (int i = 0; i < 8; ++i)
                __builtin_amdgcn_raw_buffer_store_b128(r[i], dr, vs + (i & 3) * 1024, so + (i >> 2) * 4096, AUX_SYS);
        };
        auto flag = [&](int b) {
            const bool real = (FLAG_MASK >> b) & 1;
            unsigned* const tgt = (unsigned*)((char*)scratch + (real ? fdelta + (slot + NPUSH * (b >> 4)) * 128 : 0ll));
            st_sys(tgt, fbase + (b & 15) + 1);
        };
        store_burst(d[0], 0);
        load_burst(d[0], 2);
        store_burst(d[1], 1);
        load_burst(d[1], 3);
        asm volatile("s_waitcnt vmcnt(24)" ::: "memory");
        flag(0);
        for (int b = 2; b < NBURST; b += 2) {
            if (b == NBURST / 2) {
                __builtin_amdgcn_s_barrier();
                st_agent(cnt, call);
            }
            store_burst(d[0], b);
            load_burst(d[0], b + 2);
            asm volatile("s_waitcnt vmcnt(25)" ::: "memory");
            flag(b - 1);
            store_burst(d[1], b + 1);
            load_burst(d[1], b + 3);
            asm volatile("s_waitcnt vmcnt(25)" ::: "memory");
            flag(b);
        }
        asm volatile("s_waitcnt vmcnt(9)" ::: "memory");
        flag(NBURST - 1);
        return;
    }
    if (label == 0) {
        if (slot >= 7 * NPUSH) return;
        label = 1 + slot / NPUSH;
        slot = slot % NPUSH;
    }

    lds_u32_ptr sync = (lds_u32_ptr)(lds + LDS_W);
    const int nblk = slot & 15, rg = slot >> 4;
    const unsigned cv = ld_agent(cnt);
    if (wave == 0 && lane < 8) sync[lane] = 0;
    {
        const unsigned char* wsrc = p.w + (size_t)(lane >> 1) * NLOC + nblk * 32 + (((lane & 1) ^ ((lane >> 5) & 1)) * 16);
#pragma unroll
        for (int i = 0; i < 16; ++i) {
            const int blk = i * 8 + wave;
            __builtin_amdgcn_global_load_lds((const __attribute__((address_space(1))) void*)(wsrc + (size_t)blk * 32 * NLOC),
                                             (__attribute__((address_space(3))) void*)(lds + blk * 1024), 16, 0, 0);
        }
    }
    const float scale = p.sx[0] * p.sw[0];
    const unsigned call = rfl(cv) + 1;
    unsigned* const tmo = (unsigned*)(myws + TMO_OFF);
    const unsigned fbase = (call - 1) * NCHUNK;
    __builtin_amdgcn_s_waitcnt(0x0070);
    __builtin_amdgcn_s_barrier();
    compiler_fence();
    if (threadIdx.x == 0) st_agent(cnt, call);

    const int g = lane >> 4, li = lane & 15;
    const unsigned wb0 = g * 512 + (li >> 1) * 32 + (g & 1) * 16 + (li & 1) * 8;
    f32x4 acc[2][2];
    i32x8 wfa[2], wfb[2];
    auto read_w = [&](i32x8 (&wf)[2], int n) {
        const unsigned char* wp = lds + (unsigned)n * 4096;
#pragma unroll
        for (int t = 0; t < 2; ++t)
#pragma unroll
            for (int q = 0; q < 4; ++q) {
                const i32x2 v = __builtin_amdgcn_ds_read_tr8_b64_v2i32(
                    (lds_i32x2_ptr)(wp + (wb0 ^ (t * 16)) + (q >> 1) * 2048 + (q & 1) * 256));
                wf[t][2 * q] = v[0];
                wf[t][2 * q + 1] = v[1];
            }
        __builtin_amdgcn_sched_barrier(0);
    };
    auto mma = [&](const i32x8 (&wf)[2], const u32x4 (&r)[8], int s) {
#pragma unroll
        for (int ms = 0; ms < 2; ++ms) {
            const i32x8 xf = pack8(r[(s * 2 + ms) * 2], r[(s * 2 + ms) * 2 + 1]);
#pragma unroll
            for (int t = 0; t < 2; ++t)
                acc[ms][t] = __builtin_amdgcn_mfma_scale_f32_16x16x128_f8f6f4(wf[t], xf, acc[ms][t], 1, 1, 0, 0, 0, 0);
        }
        __builtin_amdgcn_sched_barrier(0);
    };
    auto compute = [&](const u32x4 (&r)[8], int j) {
        read_w(wfb, 2 * j + 1);
        mma(wfa, r, 0);
        read_w(wfa, (2 * j + 2) & (2 * NCHUNK - 1));
        mma(wfb, r, 1);
    };
    auto clear_acc = [&]() {
#pragma unroll
        for (int i = 0; i < 4; ++i) acc[i >> 1][i & 1] = f32x4{0.f, 0.f, 0.f, 0.f};
    };
    read_w(wfa, 0);
    const auto orr = __builtin_amdgcn_make_buffer_rsrc(p.out, 0, NR * MLOC * NLOC * 4, RSRC3);
    const float nsl2e = -scale * 1.44269504088896340736f;
    const float lim = 60.f * 1.44269504088896340736f;
    auto epilogue = [&](unsigned row0, bool poison) {
        const unsigned vo = ((row0 + li) * NLOC + nblk * 32 + g * 4) * 4;
        if (poison) {
            const float nan = __builtin_nanf("");
#pragma unroll
            for (int i = 0; i < 4; ++i) acc[i >> 1][i & 1] = f32x4{nan, nan, nan, nan};
        }
#pragma unroll
        for (int ms = 0; ms < 2; ++ms)
#pragma unroll
            for (int t = 0; t < 2; ++t) {
                u32x4 v;
#pragma unroll
                for (int e = 0; e < 4; ++e) {
                    const float a = acc[ms][t][e];
                    const float ex = __builtin_amdgcn_exp2f(__builtin_amdgcn_fmed3f(a * nsl2e, -lim, lim));
                    v[e] = __builtin_bit_cast(unsigned, (a * scale) * __builtin_amdgcn_rcpf(1.f + ex));
                }
                __builtin_amdgcn_raw_buffer_store_b128(v, orr, vo + ms * 16 * NLOC * 4 + t * 64, 0, AUX_WT);
            }
    };
    u32x4 a0[8], a1[8];

    {
        const int q = (label - 1) * 2 + rg;
        const int lu = wave == 7 ? q : (wave == 6 && q < 2 ? 14 + q : -1);
        if (lu >= 0) {
            auto load_local = [&](u32x4 (&r)[8], int j) {
                load_x(r, (unsigned)lu * UNIT_BYTES + (unsigned)j * 256);
                __builtin_amdgcn_sched_barrier(0);
            };
            clear_acc();
            load_local(a0, 0);
            for (int j = 0; j < NCHUNK; j += 2) {
                load_local(a1, j + 1);
                compute(a0, j);
                load_local(a0, (j + 2) & (NCHUNK - 1));
                compute(a1, j + 1);
            }
            epilogue(rank * MLOC + lu * 32, false);
        }
    }

    const int unit = 2 * wave + rg;
    const unsigned src = (p.peers >> (4 * label)) & 7;
    const unsigned* const flagp = (const unsigned*)(myws + FLAG_OFF) + (src * 16 + unit) * 32;
    const auto ar = __builtin_amdgcn_make_buffer_rsrc(
        myws + (size_t)(call & 1) * STAGE_BYTES + (size_t)src * SLOT, 0, SLOT, RSRC3);
    const unsigned va = lane * 16;
    const unsigned ubase = (unsigned)unit * USTRIDE;
    if (wave >= NPUSH / 2)
        while (rfl(sync[wave]) == 0) __builtin_amdgcn_s_sleep(8);

    unsigned avail = 0;
    bool bad = false;
    unsigned spins = 0;
    unsigned twait = 0;
    auto expired = [&]() -> bool {
        if ((++spins & 63) != 0) return false;
        const unsigned now = rfl((unsigned)__builtin_amdgcn_s_memrealtime());
        __builtin_amdgcn_s_waitcnt(0xC07F);
        if (spins == 64) twait = now;
        if (rfl(ld_agent(tmo)) != call && now - twait <= DEADLINE_TICKS) return false;
        st_agent(tmo, call);
        bad = true;
        return true;
    };
    auto wait_chunk = [&](int j) {
        const unsigned need = fbase + j + 1;
        while (avail < need) {
            avail = rfl(ld_sys(flagp));
            if (avail < need) {
                if (expired()) avail = 0xFFFFFFFFu;
                else __builtin_amdgcn_s_sleep(2);
            }
            if (avail >= fbase + NCHUNK) sync[(wave + NPUSH / 2) & 15] = 1;
        }
    };
    auto load_chunk = [&](u32x4 (&r)[8], int j) {
        const unsigned so = ubase + (unsigned)j * CHUNK_BYTES;
        compiler_fence();
#pragma unroll
        for (int i = 0; i < 8; ++i)
            r[i] = __builtin_amdgcn_raw_buffer_load_b128(ar, va + (i & 3) * 1024, so + (i >> 2) * 4096, AUX_SYS);
        __builtin_amdgcn_sched_barrier(0);
    };
    auto run = [&](int jend) {
        wait_chunk(0);
        load_chunk(a0, 0);
        for (int j = 1; j < jend - 1; j += 2) {
            wait_chunk(j);
            load_chunk(a1, j);
            compute(a0, j - 1);
            wait_chunk(j + 1);
            load_chunk(a0, j + 1);
            compute(a1, j);
        }
        wait_chunk(jend - 1);
        load_chunk(a1, jend - 1);
    };
    clear_acc();
    if (wave < 8 - NPUSH / 2) {
        run(NCHUNK);
        const unsigned tv = ld_agent(tmo);
        compute(a0, NCHUNK - 2);
        compute(a1, NCHUNK - 1);
        bad = bad || rfl(tv) == call;
    } else {
        static_assert(NTAIL == 4, "the tail below is written for 4 chunks");
        constexpr int T0 = NCHUNK - NTAIL;
        run(T0);
        u32x4 t0[8], t1[8];
        unsigned cover = avail;
        load_chunk(t0, T0);
        load_chunk(t1, T0 + 1);
        compute(a0, T0 - 2);
        compute(a1, T0 - 1);
        auto settle = [&](u32x4 (&r)[8], int c, auto&& reload) {
            const unsigned need = fbase + T0 + c + 1;
            while (cover < need) {
                unsigned m = 0xFFFFFFFFu;
#pragma unroll
                for (int i = 0; i < 8; ++i) m = min(min(m, min(r[i][0], r[i][1])), min(r[i][2], r[i][3]));
                if (__builtin_amdgcn_ballot_w64(m == 0) == 0) break;
                cover = avail;
                const unsigned fv = ld_sys(flagp);
                reload();
                avail = rfl(fv);
                if (expired()) cover = avail = 0xFFFFFFFFu;
            }
        };
        settle(t0, 0, [&]() { load_chunk(t0, T0); load_chunk(t1, T0 + 1); });
        compute(t0, T0);
        load_chunk(a0, T0 + 2);
        settle(t1, 1, [&]() { load_chunk(t1, T0 + 1); load_chunk(a0, T0 + 2); });
        compute(t1, T0 + 1);
        load_chunk(a1, T0 + 3);
        settle(a0, 2, [&]() { load_chunk(a0, T0 + 2); load_chunk(a1, T0 + 3); });
        compute(a0, T0 + 2);
        settle(a1, 3, [&]() { load_chunk(a1, T0 + 3); });
        const unsigned tv = ld_agent(tmo);
        compute(a1, T0 + 3);
        bad = bad || rfl(tv) == call;
    }
    epilogue(src * MLOC + unit * 32, bad);
}

}

extern "C" void kernel_launch(int rank, int nranks, ncclComm_t comm, hipStream_t stream,
                              void* const* d_in, const int* in_sizes, int n_in,
                              void* d_out, int out_size,
                              const int* in_dtypes, int out_dtype,
                              void* const* d_out_all,
                              void* const* d_ws_all, size_t ws_bytes) {
    (void)comm; (void)in_dtypes; (void)out_dtype; (void)d_out_all;
    if (nranks != NR || n_in != 4 || (unsigned)rank >= (unsigned)NR || ws_bytes < WS_NEED ||
        in_sizes[0] != (int)SHARD || in_sizes[1] != KDIM * NLOC || out_size != NR * MLOC * NLOC) {
        fprintf(stderr, "unexpected problem shape\n");
        return;
    }
    Params p;
    p.x = (const unsigned char*)d_in[0];
    p.w = (const unsigned char*)d_in[1];
    p.sx = (const float*)d_in[2];
    p.sw = (const float*)d_in[3];
    p.out = (float*)d_out;
    for (int i = 0; i < NR; ++i) p.ws[i] = (unsigned char*)d_ws_all[i];
    p.rank = rank;
    {
        int dev = -1;
        const bool ident = hipGetDevice(&dev) == hipSuccess && dev == rank;
        const unsigned grp = 0xB2u;
        unsigned t = (unsigned)rank;
        int l = 1;
        for (int pass = 0; pass < 2; ++pass)
            for (int i = 1; i < NR; ++i) {
                const int q = (rank + i) & 7;
                const bool same = !ident || ((grp >> q) & 1) == ((grp >> rank) & 1);
                if (same == (pass == 0)) t |= (unsigned)q << (4 * l++);
            }
        p.peers = t;
    }
    hipLaunchKernelGGL(ag_gemm_silu_kernel, dim3(256), dim3(512), 0, stream, p);
    HIP_CHECK(hipGetLastError());
}
```

```cpp
#include <hip/hip_runtime.h>
#include <rccl/rccl.h>
#include <cstdio>
#include <cstdint>

#define HIP_CHECK(expr) do { hipError_t _e = (expr); if (_e != hipSuccess) { \
    fprintf(stderr, "HIP error %s at %s:%d\n", hipGetErrorString(_e), __FILE__, __LINE__); \
    return; } } while (0)

namespace {

constexpr int NR = 8;
constexpr int KDIM = 4096;
constexpr int MLOC = 512;
constexpr int NLOC = 512;
constexpr unsigned SHARD = MLOC * KDIM;
constexpr unsigned UNIT_BYTES = 32 * KDIM;
constexpr int NCHUNK = 16;
constexpr unsigned CHUNK_BYTES = 8192;
constexpr int NPUSH = 4;
constexpr int NBURST = 16 * NCHUNK / NPUSH;
constexpr int NTAIL = 4;
constexpr unsigned long long FLAG_MASK = 0x8faa800080008000ull;
constexpr unsigned USTRIDE = UNIT_BYTES + 4096;
constexpr unsigned SLOT = 16 * USTRIDE;
constexpr size_t STAGE_BYTES = (size_t)NR * SLOT;
constexpr unsigned TAIL_SRC = NPUSH * NTAIL * CHUNK_BYTES;
constexpr unsigned TAIL_BYTES = NR * TAIL_SRC;
constexpr size_t TAIL_OFF = 2 * STAGE_BYTES;
constexpr size_t FLAG_OFF = TAIL_OFF + 3 * TAIL_BYTES;
constexpr size_t CNT_OFF = FLAG_OFF + 8 * 16 * 128;
constexpr size_t TMO_OFF = CNT_OFF + 256 * 64;
constexpr size_t WS_NEED = TMO_OFF + 128;
constexpr unsigned DEADLINE_TICKS = 40000000u;
constexpr int LDS_W = KDIM * 32;
constexpr int AUX_SYS = 17;
constexpr int AUX_WT = 16;
constexpr unsigned RSRC3 = 0x00020000;

typedef unsigned int u32x4 __attribute__((ext_vector_type(4)));
typedef int i32x2 __attribute__((ext_vector_type(2)));
typedef int i32x8 __attribute__((ext_vector_type(8)));
typedef float f32x4 __attribute__((ext_vector_type(4)));
typedef __attribute__((address_space(3))) i32x2* lds_i32x2_ptr;
typedef __attribute__((address_space(3))) volatile unsigned* lds_u32_ptr;

struct Params {
    const unsigned char* x;
    const unsigned char* w;
    const float* sx;
    const float* sw;
    float* out;
    unsigned char* ws[NR];
    int rank;
    unsigned peers;
};

__device__ __forceinline__ unsigned ld_sys(const unsigned* p) {
    return __hip_atomic_load(p, __ATOMIC_RELAXED, __HIP_MEMORY_SCOPE_SYSTEM);
}
__device__ __forceinline__ void st_sys(unsigned* p, unsigned v) {
    __hip_atomic_store(p, v, __ATOMIC_RELAXED, __HIP_MEMORY_SCOPE_SYSTEM);
}
__device__ __forceinline__ unsigned ld_agent(const unsigned* p) {
    return __hip_atomic_load(p, __ATOMIC_RELAXED, __HIP_MEMORY_SCOPE_AGENT);
}
__device__ __forceinline__ void st_agent(unsigned* p, unsigned v) {
    __hip_atomic_store(p, v, __ATOMIC_RELAXED, __HIP_MEMORY_SCOPE_AGENT);
}
__device__ __forceinline__ unsigned rfl(unsigned v) { return __builtin_amdgcn_readfirstlane(v); }
__device__ __forceinline__ void compiler_fence() { asm volatile("" ::: "memory"); }

__device__ __forceinline__ i32x8 pack8(u32x4 lo, u32x4 hi) {
    i32x8 r;
    r[0] = lo[0]; r[1] = lo[1]; r[2] = lo[2]; r[3] = lo[3];
    r[4] = hi[0]; r[5] = hi[1]; r[6] = hi[2]; r[7] = hi[3];
    return r;
}

__global__ void __launch_bounds__(512)
ag_gemm_silu_kernel(Params p) {
    __shared__ __attribute__((aligned(16))) unsigned char lds[LDS_W + 64];
    const int lane = threadIdx.x & 63;
    const int wave = rfl(threadIdx.x >> 6);
    const int bid = blockIdx.x;
    const unsigned rank = (unsigned)p.rank & 7;
    unsigned char* const myws = p.ws[0];
    unsigned* const cnt = (unsigned*)(myws + CNT_OFF) + bid * 16;
    const auto xr = __builtin_amdgcn_make_buffer_rsrc((void*)p.x, 0, SHARD, RSRC3);
    const unsigned vx0 = (lane & 15) * KDIM + (lane >> 4) * 16;
    const unsigned vx1 = vx0 + 16 * KDIM;
    auto load_x = [&](u32x4 (&r)[8], unsigned so) {
#pragma unroll
        for (int s = 0; s < 2; ++s)
#pragma unroll
            for (int ms = 0; ms < 2; ++ms)
#pragma unroll
                for (int h = 0; h < 2; ++h)
                    r[(s * 2 + ms) * 2 + h] =
                        __builtin_amdgcn_raw_buffer_load_b128(xr, (ms ? vx1 : vx0) + s * 128 + h * 64, so, 0);
    };

    int label = bid & 7, slot = bid >> 3;
    if (__builtin_expect(label != 0 && slot < NPUSH, 1)) {
        if (wave != 0) return;
        const unsigned cv = ld_agent(cnt);
        u32x4 d[2][8];
        auto load_burst = [&](u32x4 (&r)[8], int b) {
            b &= NBURST - 1;
            load_x(r, (unsigned)(slot + NPUSH * (b >> 4)) * UNIT_BYTES + (unsigned)(b & 15) * 256);
        };
        load_burst(d[0], 0);
        load_burst(d[1], 1);
        const unsigned call = rfl(cv) + 1;
        unsigned char* const dws = p.ws[label];
        const auto dr = __builtin_amdgcn_make_buffer_rsrc(dws, 0, FLAG_OFF, RSRC3);
        const unsigned sbase = (call & 1) * (unsigned)STAGE_BYTES + rank * SLOT + slot * USTRIDE;
        const unsigned tbase = (unsigned)TAIL_OFF + (call % 3) * TAIL_BYTES + rank * TAIL_SRC +
                               slot * (NTAIL * CHUNK_BYTES) - (NBURST - NTAIL) * CHUNK_BYTES;
        unsigned* const flags = (unsigned*)(dws + FLAG_OFF) + rank * 16 * 32;
        const unsigned fbase = (call - 1) * NCHUNK;
        unsigned* const scratch = cnt + 8;
        const long long fdelta = (char*)flags - (char*)scratch;
        const unsigned vs = lane * 16;
        auto store_burst = [&](const u32x4 (&r)[8], int b) {
            const unsigned so = b < NBURST - NTAIL
                ? sbase + (unsigned)(b >> 4) * (NPUSH * USTRIDE) + (unsigned)(b & 15) * CHUNK_BYTES
                : tbase + (unsigned)b * CHUNK_BYTES;
#pragma unroll
            for (int i = 0; i < 8; ++i)
                __builtin_amdgcn_raw_buffer_store_b128(r[i], dr, vs + (i & 3) * 1024, so + (i >> 2) * 4096, AUX_SYS);
        };
        auto flag = [&](int b) {
            const bool real = (FLAG_MASK >> b) & 1;
            unsigned* const tgt = (unsigned*)((char*)scratch + (real ? fdelta + (slot + NPUSH * (b >> 4)) * 128 : 0ll));
            st_sys(tgt, fbase + (b & 15) + 1);
        };
        store_burst(d[0], 0);
        load_burst(d[0], 2);
        store_burst(d[1], 1);
        load_burst(d[1], 3);
        asm volatile("s_waitcnt vmcnt(24)" ::: "memory");
        flag(0);
        for (int b = 2; b < NBURST; b += 2) {
            store_burst(d[0], b);
            load_burst(d[0], b + 2);
            asm volatile("s_waitcnt vmcnt(25)" ::: "memory");
            flag(b - 1);
            store_burst(d[1], b + 1);
            load_burst(d[1], b + 3);
            asm volatile("s_waitcnt vmcnt(25)" ::: "memory");
            flag(b);
        }
        asm volatile("s_waitcnt vmcnt(9)" ::: "memory");
        flag(NBURST - 1);
        st_agent(cnt, call);
        return;
    }
    if (label == 0) {
        if (slot >= 7 * NPUSH) {
            const unsigned call = rfl(ld_agent(cnt)) + 1;
            const auto zr = __builtin_amdgcn_make_buffer_rsrc(
                myws + TAIL_OFF + (size_t)((call + 2) % 3) * TAIL_BYTES, 0, TAIL_BYTES, RSRC3);
            const unsigned zo = (unsigned)(slot - 7 * NPUSH) * (TAIL_BYTES / 4) + threadIdx.x * 16;
#pragma unroll 4
            for (unsigned i = 0; i < TAIL_BYTES / 4; i += 8192)
                __builtin_amdgcn_raw_buffer_store_b128(u32x4{0u, 0u, 0u, 0u}, zr, zo + i, 0, AUX_SYS);
            asm volatile("s_waitcnt vmcnt(0)" ::: "memory");
            __builtin_amdgcn_s_barrier();
            if (threadIdx.x == 0) st_agent(cnt, call);
            return;
        }
        label = 1 + slot / NPUSH;
        slot = slot % NPUSH;
    }

    lds_u32_ptr sync = (lds_u32_ptr)(lds + LDS_W);
    const int nblk = slot & 15, rg = slot >> 4;
    const unsigned cv = ld_agent(cnt);
    if (wave == 0 && lane < 8) sync[lane] = 0;
    {
        const unsigned char* wsrc = p.w + (size_t)(lane >> 1) * NLOC + nblk * 32 + (((lane & 1) ^ ((lane >> 5) & 1)) * 16);
#pragma unroll
        for (int i = 0; i < 16; ++i) {
            const int blk = i * 8 + wave;
            __builtin_amdgcn_global_load_lds((const __attribute__((address_space(1))) void*)(wsrc + (size_t)blk * 32 * NLOC),
                                             (__attribute__((address_space(3))) void*)(lds + blk * 1024), 16, 0, 0);
        }
    }
    const float scale = p.sx[0] * p.sw[0];
    const unsigned call = rfl(cv) + 1;
    unsigned* const tmo = (unsigned*)(myws + TMO_OFF);
    const unsigned fbase = (call - 1) * NCHUNK;
    __builtin_amdgcn_s_waitcnt(0x0070);
    __builtin_amdgcn_s_barrier();
    compiler_fence();
    if (threadIdx.x == 0) st_agent(cnt, call);

    const int g = lane >> 4, li = lane & 15;
    const unsigned wb0 = g * 512 + (li >> 1) * 32 + (g & 1) * 16 + (li & 1) * 8;
    f32x4 acc[2][2];
    i32x8 wfa[2], wfb[2];
    auto read_w = [&](i32x8 (&wf)[2], int n) {
        const unsigned char* wp = lds + (unsigned)n * 4096;
#pragma unroll
        for (int t = 0; t < 2; ++t)
#pragma unroll
            for (int q = 0; q < 4; ++q) {
                const i32x2 v = __builtin_amdgcn_ds_read_tr8_b64_v2i32(
                    (lds_i32x2_ptr)(wp + (wb0 ^ (t * 16)) + (q >> 1) * 2048 + (q & 1) * 256));
                wf[t][2 * q] = v[0];
                wf[t][2 * q + 1] = v[1];
            }
        __builtin_amdgcn_sched_barrier(0);
    };
    auto mma = [&](const i32x8 (&wf)[2], const u32x4 (&r)[8], int s) {
#pragma unroll
        for (int ms = 0; ms < 2; ++ms) {
            const i32x8 xf = pack8(r[(s * 2 + ms) * 2], r[(s * 2 + ms) * 2 + 1]);
#pragma unroll
            for (int t = 0; t < 2; ++t)
                acc[ms][t] = __builtin_amdgcn_mfma_scale_f32_16x16x128_f8f6f4(wf[t], xf, acc[ms][t], 1, 1, 0, 0, 0, 0);
        }
        __builtin_amdgcn_sched_barrier(0);
    };
    auto compute = [&](const u32x4 (&r)[8], int j) {
        read_w(wfb, 2 * j + 1);
        mma(wfa, r, 0);
        read_w(wfa, (2 * j + 2) & (2 * NCHUNK - 1));
        mma(wfb, r, 1);
    };
    auto clear_acc = [&]() {
#pragma unroll
        for (int i = 0; i < 4; ++i) acc[i >> 1][i & 1] = f32x4{0.f, 0.f, 0.f, 0.f};
    };
    read_w(wfa, 0);
    const auto orr = __builtin_amdgcn_make_buffer_rsrc(p.out, 0, NR * MLOC * NLOC * 4, RSRC3);
    const float nsl2e = -scale * 1.44269504088896340736f;
    const float lim = 60.f * 1.44269504088896340736f;
    auto epilogue = [&](unsigned row0, bool poison) {
        const unsigned vo = ((row0 + li) * NLOC + nblk * 32 + g * 4) * 4;
        if (poison) {
            const float nan = __builtin_nanf("");
#pragma unroll
            for (int i = 0; i < 4; ++i) acc[i >> 1][i & 1] = f32x4{nan, nan, nan, nan};
        }
#pragma unroll
        for (int ms = 0; ms < 2; ++ms)
#pragma unroll
            for (int t = 0; t < 2; ++t) {
                u32x4 v;
#pragma unroll
                for (int e = 0; e < 4; ++e) {
                    const float a = acc[ms][t][e];
                    const float ex = __builtin_amdgcn_exp2f(__builtin_amdgcn_fmed3f(a * nsl2e, -lim, lim));
                    v[e] = __builtin_bit_cast(unsigned, (a * scale) * __builtin_amdgcn_rcpf(1.f + ex));
                }
                __builtin_amdgcn_raw_buffer_store_b128(v, orr, vo + ms * 16 * NLOC * 4 + t * 64, 0, AUX_WT);
            }
    };
    u32x4 a0[8], a1[8];

    {
        const int q = (label - 1) * 2 + rg;
        const int lu = wave == 7 ? q : (wave == 6 && q < 2 ? 14 + q : -1);
        if (lu >= 0) {
            auto load_local = [&](u32x4 (&r)[8], int j) {
                load_x(r, (unsigned)lu * UNIT_BYTES + (unsigned)j * 256);
                __builtin_amdgcn_sched_barrier(0);
            };
            clear_acc();
            load_local(a0, 0);
            for (int j = 0; j < NCHUNK; j += 2) {
                load_local(a1, j + 1);
                compute(a0, j);
                load_local(a0, (j + 2) & (NCHUNK - 1));
                compute(a1, j + 1);
            }
            epilogue(rank * MLOC + lu * 32, false);
        }
    }

    const int unit = 2 * wave + rg;
    const unsigned src = (p.peers >> (4 * label)) & 7;
    const unsigned* const flagp = (const unsigned*)(myws + FLAG_OFF) + (src * 16 + unit) * 32;
    const auto ar = __builtin_amdgcn_make_buffer_rsrc(myws, 0, FLAG_OFF, RSRC3);
    const unsigned va = lane * 16;
    const unsigned ubase = (call & 1) * (unsigned)STAGE_BYTES + src * SLOT + (unsigned)unit * USTRIDE;
    const unsigned tbase = (unsigned)TAIL_OFF + (call % 3) * TAIL_BYTES + src * TAIL_SRC +
                           (unsigned)(unit & (NPUSH - 1)) * (NTAIL * CHUNK_BYTES);
    if (wave >= NPUSH / 2)
        while (rfl(sync[wave]) == 0) __builtin_amdgcn_s_sleep(8);

    unsigned avail = 0;
    bool bad = false;
    unsigned spins = 0;
    unsigned twait = 0;
    auto expired = [&]() -> bool {
        if ((++spins & 63) != 0) return false;
        const unsigned now = rfl((unsigned)__builtin_amdgcn_s_memrealtime());
        __builtin_amdgcn_s_waitcnt(0xC07F);
        if (spins == 64) twait = now;
        if (rfl(ld_agent(tmo)) != call && now - twait <= DEADLINE_TICKS) return false;
        st_agent(tmo, call);
        bad = true;
        return true;
    };
    auto wait_chunk = [&](int j) {
        const unsigned need = fbase + j + 1;
        while (avail < need) {
            avail = rfl(ld_sys(flagp));
            if (avail < need) {
                if (expired()) avail = 0xFFFFFFFFu;
                else __builtin_amdgcn_s_sleep(2);
            }
            if (avail >= fbase + NCHUNK) sync[(wave + NPUSH / 2) & 15] = 1;
        }
    };
    auto load_at = [&](u32x4 (&r)[8], unsigned so) {
        compiler_fence();
#pragma unroll
        for (int i = 0; i < 8; ++i)
            r[i] = __builtin_amdgcn_raw_buffer_load_b128(ar, va + (i & 3) * 1024, so + (i >> 2) * 4096, AUX_SYS);
        __builtin_amdgcn_sched_barrier(0);
    };
    auto load_chunk = [&](u32x4 (&r)[8], int j) { load_at(r, ubase + (unsigned)j * CHUNK_BYTES); };
    auto load_tail = [&](u32x4 (&r)[8], int c) { load_at(r, tbase + (unsigned)c * CHUNK_BYTES); };
    auto run = [&](int jend) {
        wait_chunk(0);
        load_chunk(a0, 0);
        for (int j = 1; j < jend - 1; j += 2) {
            wait_chunk(j);
            load_chunk(a1, j);
            compute(a0, j - 1);
            wait_chunk(j + 1);
            load_chunk(a0, j + 1);
            compute(a1, j);
        }
        wait_chunk(jend - 1);
        load_chunk(a1, jend - 1);
    };
    clear_acc();
    if (wave < 8 - NPUSH / 2) {
        run(NCHUNK);
        const unsigned tv = ld_agent(tmo);
        compute(a0, NCHUNK - 2);
        compute(a1, NCHUNK - 1);
        bad = bad || rfl(tv) == call;
    } else {
        static_assert(NTAIL == 4, "the tail below is written for 4 chunks");
        constexpr int T0 = NCHUNK - NTAIL;
        run(T0);
        u32x4 t0[8], t1[8];
        unsigned cover = avail;
        load_tail(t0, 0);
        load_tail(t1, 1);
        compute(a0, T0 - 2);
        compute(a1, T0 - 1);
        auto settle = [&](u32x4 (&r)[8], int c, auto&& reload) {
            const unsigned need = fbase + T0 + c + 1;
            while (cover < need) {
                unsigned m = 0xFFFFFFFFu;
#pragma unroll
                for (int i = 0; i < 8; ++i) m = min(min(m, min(r[i][0], r[i][1])), min(r[i][2], r[i][3]));
                if (__builtin_amdgcn_ballot_w64(m == 0) == 0) break;
                cover = avail;
                const unsigned fv = ld_sys(flagp);
                reload();
                avail = rfl(fv);
                if (expired()) cover = avail = 0xFFFFFFFFu;
            }
        };
        settle(t0, 0, [&]() { load_tail(t0, 0); load_tail(t1, 1); });
        compute(t0, T0);
        load_tail(a0, 2);
        settle(t1, 1, [&]() { load_tail(t1, 1); load_tail(a0, 2); });
        compute(t1, T0 + 1);
        load_tail(a1, 3);
        settle(a0, 2, [&]() { load_tail(a0, 2); load_tail(a1, 3); });
        compute(a0, T0 + 2);
        settle(a1, 3, [&]() { load_tail(a1, 3); });
        const unsigned tv = ld_agent(tmo);
        compute(a1, T0 + 3);
        bad = bad || rfl(tv) == call;
    }
    epilogue(src * MLOC + unit * 32, bad);
}

}

extern "C" void kernel_launch(int rank, int nranks, ncclComm_t comm, hipStream_t stream,
                              void* const* d_in, const int* in_sizes, int n_in,
                              void* d_out, int out_size,
                              const int* in_dtypes, int out_dtype,
                              void* const* d_out_all,
                              void* const* d_ws_all, size_t ws_bytes) {
    (void)comm; (void)in_dtypes; (void)out_dtype; (void)d_out_all;
    if (nranks != NR || n_in != 4 || (unsigned)rank >= (unsigned)NR || ws_bytes < WS_NEED ||
        in_sizes[0] != (int)SHARD || in_sizes[1] != KDIM * NLOC || out_size != NR * MLOC * NLOC) {
        fprintf(stderr, "unexpected problem shape\n");
        return;
    }
    Params p;
    p.x = (const unsigned char*)d_in[0];
    p.w = (const unsigned char*)d_in[1];
    p.sx = (const float*)d_in[2];
    p.sw = (const float*)d_in[3];
    p.out = (float*)d_out;
    p.rank = rank;
    {
        int dev = -1;
        const bool ident = hipGetDevice(&dev) == hipSuccess && dev == rank;
        const unsigned grp = 0xB2u;
        unsigned t = (unsigned)rank;
        int l = 1;
        for (int pass = 0; pass < 2; ++pass)
            for (int i = 1; i < NR; ++i) {
                const int q = (rank + i) & 7;
                const bool same = !ident || ((grp >> q) & 1) == ((grp >> rank) & 1);
                if (same == (pass == 0)) t |= (unsigned)q << (4 * l++);
            }
        p.peers = t;
        for (int i = 0; i < NR; ++i) p.ws[i] = (unsigned char*)d_ws_all[(t >> (4 * i)) & 7];
    }
    hipLaunchKernelGGL(ag_gemm_silu_kernel, dim3(256), dim3(512), 0, stream, p);
    HIP_CHECK(hipGetLastError());
}
```

```cpp
#include <hip/hip_runtime.h>
#include <rccl/rccl.h>
#include <cstdio>
#include <cstdint>

#define HIP_CHECK(expr) do { hipError_t _e = (expr); if (_e != hipSuccess) { \
    fprintf(stderr, "HIP error %s at %s:%d\n", hipGetErrorString(_e), __FILE__, __LINE__); \
    return; } } while (0)

namespace {

constexpr int NR = 8;
constexpr int KDIM = 4096;
constexpr int MLOC = 512;
constexpr int NLOC = 512;
constexpr unsigned SHARD = MLOC * KDIM;
constexpr unsigned UNIT_BYTES = 32 * KDIM;
constexpr int NCHUNK = 16;
constexpr unsigned CHUNK_BYTES = 8192;
constexpr int NPUSH = 4;
constexpr int NBURST = 16 * NCHUNK / NPUSH;
constexpr int NTAIL = 4;
constexpr unsigned long long FLAG_MASK = 0x8faa800080008000ull;
constexpr unsigned USTRIDE = UNIT_BYTES + 4096;
constexpr unsigned SLOT = 16 * USTRIDE;
constexpr size_t STAGE_BYTES = (size_t)NR * SLOT;
constexpr unsigned TAIL_SRC = NPUSH * NTAIL * CHUNK_BYTES;
constexpr unsigned TAIL_BYTES = NR * TAIL_SRC;
constexpr size_t TAIL_OFF = 2 * STAGE_BYTES;
constexpr size_t FLAG_OFF = TAIL_OFF + 3 * TAIL_BYTES;
constexpr size_t CNT_OFF = FLAG_OFF + 8 * 16 * 128;
constexpr size_t TMO_OFF = CNT_OFF + 256 * 64;
constexpr size_t WS_NEED = TMO_OFF + 128;
constexpr unsigned DEADLINE_TICKS = 40000000u;
constexpr int LDS_W = KDIM * 32;
constexpr int AUX_SYS = 17;
constexpr int AUX_WT = 16;
constexpr unsigned RSRC3 = 0x00020000;

typedef unsigned int u32x4 __attribute__((ext_vector_type(4)));
typedef int i32x2 __attribute__((ext_vector_type(2)));
typedef int i32x8 __attribute__((ext_vector_type(8)));
typedef float f32x4 __attribute__((ext_vector_type(4)));
typedef __attribute__((address_space(3))) i32x2* lds_i32x2_ptr;
typedef __attribute__((address_space(3))) volatile unsigned* lds_u32_ptr;

struct Params {
    const unsigned char* x;
    const unsigned char* w;
    const float* sx;
    const float* sw;
    float* out;
    unsigned char* ws[NR];
    int rank;
    unsigned peers;
};

__device__ __forceinline__ unsigned ld_sys(const unsigned* p) {
    return __hip_atomic_load(p, __ATOMIC_RELAXED, __HIP_MEMORY_SCOPE_SYSTEM);
}
__device__ __forceinline__ void st_sys(unsigned* p, unsigned v) {
    __hip_atomic_store(p, v, __ATOMIC_RELAXED, __HIP_MEMORY_SCOPE_SYSTEM);
}
__device__ __forceinline__ unsigned ld_agent(const unsigned* p) {
    return __hip_atomic_load(p, __ATOMIC_RELAXED, __HIP_MEMORY_SCOPE_AGENT);
}
__device__ __forceinline__ void st_agent(unsigned* p, unsigned v) {
    __hip_atomic_store(p, v, __ATOMIC_RELAXED, __HIP_MEMORY_SCOPE_AGENT);
}
__device__ __forceinline__ unsigned rfl(unsigned v) { return __builtin_amdgcn_readfirstlane(v); }
__device__ __forceinline__ void compiler_fence() { asm volatile("" ::: "memory"); }

__device__ __forceinline__ i32x8 pack8(u32x4 lo, u32x4 hi) {
    i32x8 r;
    r[0] = lo[0]; r[1] = lo[1]; r[2] = lo[2]; r[3] = lo[3];
    r[4] = hi[0]; r[5] = hi[1]; r[6] = hi[2]; r[7] = hi[3];
    return r;
}

__global__ void __launch_bounds__(512)
ag_gemm_silu_kernel(Params p) {
    __shared__ __attribute__((aligned(16))) unsigned char lds[LDS_W + 64];
    const int lane = threadIdx.x & 63;
    const int wave = rfl(threadIdx.x >> 6);
    const int bid = blockIdx.x;
    const unsigned rank = (unsigned)p.rank & 7;
    unsigned char* const myws = p.ws[0];
    unsigned* const cnt = (unsigned*)(myws + CNT_OFF) + bid * 16;
    const auto xr = __builtin_amdgcn_make_buffer_rsrc((void*)p.x, 0, SHARD, RSRC3);
    const unsigned vx0 = (lane & 15) * KDIM + (lane >> 4) * 16;
    const unsigned vx1 = vx0 + 16 * KDIM;
    auto load_x = [&](u32x4 (&r)[8], unsigned so) {
#pragma unroll
        for (int s = 0; s < 2; ++s)
#pragma unroll
            for (int ms = 0; ms < 2; ++ms)
#pragma unroll
                for (int h = 0; h < 2; ++h)
                    r[(s * 2 + ms) * 2 + h] =
                        __builtin_amdgcn_raw_buffer_load_b128(xr, (ms ? vx1 : vx0) + s * 128 + h * 64, so, 0);
    };

    int label = bid & 7, slot = bid >> 3;
    if (__builtin_expect(label != 0 && slot < NPUSH, 1)) {
        if (wave != 0) return;
        const unsigned cv = ld_agent(cnt);
        u32x4 d[2][8];
        auto load_burst = [&](u32x4 (&r)[8], int b) {
            b &= NBURST - 1;
            load_x(r, (unsigned)(slot + NPUSH * (b >> 4)) * UNIT_BYTES + (unsigned)(b & 15) * 256);
        };
        load_burst(d[0], 0);
        load_burst(d[1], 1);
        const unsigned call = rfl(cv) + 1;
        unsigned char* const dws = p.ws[label];
        const auto dr = __builtin_amdgcn_make_buffer_rsrc(dws, 0, FLAG_OFF, RSRC3);
        const unsigned sbase = (call & 1) * (unsigned)STAGE_BYTES + rank * SLOT + slot * USTRIDE;
        const unsigned tbase = (unsigned)TAIL_OFF + (call % 3) * TAIL_BYTES + rank * TAIL_SRC +
                               slot * (NTAIL * CHUNK_BYTES) - (NBURST - NTAIL) * CHUNK_BYTES;
        unsigned* const flags = (unsigned*)(dws + FLAG_OFF) + rank * 16 * 32;
        const unsigned fbase = (call - 1) * NCHUNK;
        unsigned* const scratch = cnt + 8;
        const long long fdelta = (char*)flags - (char*)scratch;
        const unsigned vs = lane * 16;
        auto store_burst = [&](const u32x4 (&r)[8], int b) {
            const unsigned so = b < NBURST - NTAIL
                ? sbase + (unsigned)(b >> 4) * (NPUSH * USTRIDE) + (unsigned)(b & 15) * CHUNK_BYTES
                : tbase + (unsigned)b * CHUNK_BYTES;
#pragma unroll
            for (int i = 0; i < 8; ++i)
                __builtin_amdgcn_raw_buffer_store_b128(r[i], dr, vs + (i & 3) * 1024, so + (i >> 2) * 4096, AUX_SYS);
        };
        auto flag = [&](int b) {
            const bool real = (FLAG_MASK >> b) & 1;
            unsigned* const tgt = (unsigned*)((char*)scratch + (real ? fdelta + (slot + NPUSH * (b >> 4)) * 128 : 0ll));
            st_sys(tgt, fbase + (b & 15) + 1);
        };
        store_burst(d[0], 0);
        load_burst(d[0], 2);
        store_burst(d[1], 1);
        load_burst(d[1], 3);
        asm volatile("s_waitcnt vmcnt(24)" ::: "memory");
        flag(0);
        st_agent(cnt, call);
        for (int b = 2; b < NBURST; b += 2) {
            store_burst(d[0], b);
            load_burst(d[0], b + 2);
            asm volatile("s_waitcnt vmcnt(25)" ::: "memory");
            flag(b - 1);
            store_burst(d[1], b + 1);
            load_burst(d[1], b + 3);
            asm volatile("s_waitcnt vmcnt(25)" ::: "memory");
            flag(b);
        }
        asm volatile("s_waitcnt vmcnt(9)" ::: "memory");
        flag(NBURST - 1);
        return;
    }
    if (label == 0) {
        if (slot >= 7 * NPUSH) {
            const unsigned call = rfl(ld_agent(cnt)) + 1;
            const auto zr = __builtin_amdgcn_make_buffer_rsrc(
                myws + TAIL_OFF + (size_t)((call + 2) % 3) * TAIL_BYTES, 0, TAIL_BYTES, RSRC3);
            const unsigned zo = (unsigned)(slot - 7 * NPUSH) * (TAIL_BYTES / 4) + threadIdx.x * 16;
#pragma unroll 4
            for (unsigned i = 0; i < TAIL_BYTES / 4; i += 8192)
                __builtin_amdgcn_raw_buffer_store_b128(u32x4{0u, 0u, 0u, 0u}, zr, zo + i, 0, AUX_SYS);
            asm volatile("s_waitcnt vmcnt(0)" ::: "memory");
            __builtin_amdgcn_s_barrier();
            if (threadIdx.x == 0) st_agent(cnt, call);
            return;
        }
        label = 1 + slot / NPUSH;
        slot = slot % NPUSH;
    }

    lds_u32_ptr sync = (lds_u32_ptr)(lds + LDS_W);
    const int nblk = slot & 15, rg = slot >> 4;
    const unsigned cv = ld_agent(cnt);
    if (wave == 0 && lane < 8) sync[lane] = 0;
    {
        const unsigned char* wsrc = p.w + (size_t)(lane >> 1) * NLOC + nblk * 32 + (((lane & 1) ^ ((lane >> 5) & 1)) * 16);
#pragma unroll
        for (int i = 0; i < 16; ++i) {
            const int blk = i * 8 + wave;
            __builtin_amdgcn_global_load_lds((const __attribute__((address_space(1))) void*)(wsrc + (size_t)blk * 32 * NLOC),
                                             (__attribute__((address_space(3))) void*)(lds + blk * 1024), 16, 0, 0);
        }
    }
    const float scale = p.sx[0] * p.sw[0];
    const unsigned call = rfl(cv) + 1;
    unsigned* const tmo = (unsigned*)(myws + TMO_OFF);
    const unsigned fbase = (call - 1) * NCHUNK;
    __builtin_amdgcn_s_waitcnt(0x0070);
    __builtin_amdgcn_s_barrier();
    compiler_fence();
    if (threadIdx.x == 0) st_agent(cnt, call);

    const int g = lane >> 4, li = lane & 15;
    const unsigned wb0 = g * 512 + (li >> 1) * 32 + (g & 1) * 16 + (li & 1) * 8;
    f32x4 acc[2][2];
    i32x8 wfa[2], wfb[2];
    auto read_w = [&](i32x8 (&wf)[2], int n) {
        const unsigned char* wp = lds + (unsigned)n * 4096;
#pragma unroll
        for (int t = 0; t < 2; ++t)
#pragma unroll
            for (int q = 0; q < 4; ++q) {
                const i32x2 v = __builtin_amdgcn_ds_read_tr8_b64_v2i32(
                    (lds_i32x2_ptr)(wp + (wb0 ^ (t * 16)) + (q >> 1) * 2048 + (q & 1) * 256));
                wf[t][2 * q] = v[0];
                wf[t][2 * q + 1] = v[1];
            }
        __builtin_amdgcn_sched_barrier(0);
    };
    auto mma = [&](const i32x8 (&wf)[2], const auto& r, int s) {
        constexpr int NM = sizeof(r) / sizeof(r[0]) / 4;
#pragma unroll
        for (int ms = 0; ms < NM; ++ms) {
            const i32x8 xf = pack8(r[(s * NM + ms) * 2], r[(s * NM + ms) * 2 + 1]);
#pragma unroll
            for (int t = 0; t < 2; ++t)
                acc[ms][t] = __builtin_amdgcn_mfma_scale_f32_16x16x128_f8f6f4(wf[t], xf, acc[ms][t], 1, 1, 0, 0, 0, 0);
        }
        __builtin_amdgcn_sched_barrier(0);
    };
    auto compute = [&](const auto& r, int j) {
        read_w(wfb, 2 * j + 1);
        mma(wfa, r, 0);
        read_w(wfa, (2 * j + 2) & (2 * NCHUNK - 1));
        mma(wfb, r, 1);
    };
    auto clear_acc = [&]() {
#pragma unroll
        for (int i = 0; i < 4; ++i) acc[i >> 1][i & 1] = f32x4{0.f, 0.f, 0.f, 0.f};
    };
    read_w(wfa, 0);
    const auto orr = __builtin_amdgcn_make_buffer_rsrc(p.out, 0, NR * MLOC * NLOC * 4, RSRC3);
    const float nsl2e = -scale * 1.44269504088896340736f;
    const float lim = 60.f * 1.44269504088896340736f;
    auto epilogue = [&](int nm, unsigned row0, bool poison) {
        const unsigned vo = ((row0 + li) * NLOC + nblk * 32 + g * 4) * 4;
        if (poison) {
            const float nan = __builtin_nanf("");
#pragma unroll
            for (int i = 0; i < 4; ++i) acc[i >> 1][i & 1] = f32x4{nan, nan, nan, nan};
        }
#pragma unroll
        for (int ms = 0; ms < nm; ++ms)
#pragma unroll
            for (int t = 0; t < 2; ++t) {
                u32x4 v;
#pragma unroll
                for (int e = 0; e < 4; ++e) {
                    const float a = acc[ms][t][e];
                    const float ex = __builtin_amdgcn_exp2f(__builtin_amdgcn_fmed3f(a * nsl2e, -lim, lim));
                    v[e] = __builtin_bit_cast(unsigned, (a * scale) * __builtin_amdgcn_rcpf(1.f + ex));
                }
                __builtin_amdgcn_raw_buffer_store_b128(v, orr, vo + ms * 16 * NLOC * 4 + t * 64, 0, AUX_WT);
            }
    };

    {
        const int q = (label - 1) * 2 + rg;
        const int lu = wave == 7 ? q : (wave == 6 && q < 2 ? 14 + q : -1);
        if (lu >= 0) {
            auto load_local = [&](u32x4 (&r)[8], int j) {
                load_x(r, (unsigned)lu * UNIT_BYTES + (unsigned)j * 256);
                __builtin_amdgcn_sched_barrier(0);
            };
            u32x4 a0[8], a1[8];
            clear_acc();
            load_local(a0, 0);
            for (int j = 0; j < NCHUNK; j += 2) {
                load_local(a1, j + 1);
                compute(a0, j);
                load_local(a0, (j + 2) & (NCHUNK - 1));
                compute(a1, j + 1);
            }
            epilogue(2, rank * MLOC + lu * 32, false);
        }
    }

    const unsigned src = (p.peers >> (4 * label)) & 7;
    const auto ar = __builtin_amdgcn_make_buffer_rsrc(myws, 0, FLAG_OFF, RSRC3);
    const unsigned va = lane * 16;
    const unsigned* flagp;
    unsigned ubase;
    unsigned avail;
    bool bad = false;
    unsigned spins = 0;
    unsigned twait = 0;
    auto begin = [&](int unit, int wake) {
        flagp = (const unsigned*)(myws + FLAG_OFF) + (src * 16 + unit) * 32;
        ubase = (call & 1) * (unsigned)STAGE_BYTES + src * SLOT + (unsigned)unit * USTRIDE;
        avail = 0;
        clear_acc();
        if (wake >= NPUSH / 2)
            while (rfl(sync[wake]) == 0) __builtin_amdgcn_s_sleep(8);
    };
    auto expired = [&]() -> bool {
        if ((++spins & 63) != 0) return false;
        const unsigned now = rfl((unsigned)__builtin_amdgcn_s_memrealtime());
        __builtin_amdgcn_s_waitcnt(0xC07F);
        if (spins == 64) twait = now;
        if (rfl(ld_agent(tmo)) != call && now - twait <= DEADLINE_TICKS) return false;
        st_agent(tmo, call);
        bad = true;
        return true;
    };
    auto wait_chunk = [&](int j) {
        const unsigned need = fbase + j + 1;
        while (avail < need) {
            avail = rfl(ld_sys(flagp));
            if (avail < need) {
                if (expired()) avail = 0xFFFFFFFFu;
                else __builtin_amdgcn_s_sleep(2);
            }
            if (avail >= fbase + NCHUNK) sync[(wave + NPUSH / 2) & 15] = 1;
        }
    };
    auto load_at = [&](auto& r, unsigned so) {
        constexpr int NB = sizeof(r) / sizeof(r[0]);
        compiler_fence();
#pragma unroll
        for (int i = 0; i < NB; ++i)
            r[i] = __builtin_amdgcn_raw_buffer_load_b128(ar, va + (i & (NB / 2 - 1)) * 1024, so + (i / (NB / 2)) * 4096, AUX_SYS);
        __builtin_amdgcn_sched_barrier(0);
    };
    auto run = [&](auto& a0, auto& a1, int jend) {
        wait_chunk(0);
        load_at(a0, ubase);
        for (int j = 1; j < jend - 1; j += 2) {
            wait_chunk(j);
            load_at(a1, ubase + (unsigned)j * CHUNK_BYTES);
            compute(a0, j - 1);
            wait_chunk(j + 1);
            load_at(a0, ubase + (unsigned)(j + 1) * CHUNK_BYTES);
            compute(a1, j);
        }
        wait_chunk(jend - 1);
        load_at(a1, ubase + (unsigned)(jend - 1) * CHUNK_BYTES);
    };

    if (wave < 8 - NPUSH / 2) {
        const int unit = 2 * wave + rg;
        u32x4 a0[8], a1[8];
        begin(unit, wave);
        run(a0, a1, NCHUNK);
        const unsigned tv = ld_agent(tmo);
        compute(a0, NCHUNK - 2);
        compute(a1, NCHUNK - 1);
        epilogue(2, src * MLOC + unit * 32, bad || rfl(tv) == call);
        if (wave >= NPUSH / 2) return;
    }
    {
        static_assert(NTAIL == 4 && NPUSH == 4, "the tail below is written for 4 chunks of 4 units");
        constexpr int T0 = NCHUNK - NTAIL;
        const int unit = 12 + 2 * (wave & 1) + rg;
        const unsigned half = wave < 2;
        u32x4 a0[4], a1[4], t0[4], t1[4];
        begin(unit, 6 + (wave & 1));
        ubase += half * 2048;
        const unsigned tbase = (unsigned)TAIL_OFF + (call % 3) * TAIL_BYTES + src * TAIL_SRC +
                               (unsigned)(unit & (NPUSH - 1)) * (NTAIL * CHUNK_BYTES) + half * 2048;
        auto load_tail = [&](u32x4 (&r)[4], int c) { load_at(r, tbase + (unsigned)c * CHUNK_BYTES); };
        run(a0, a1, T0);
        unsigned cover = avail;
        load_tail(t0, 0);
        load_tail(t1, 1);
        compute(a0, T0 - 2);
        compute(a1, T0 - 1);
        auto settle = [&](u32x4 (&r)[4], int c, auto&& reload) {
            const unsigned need = fbase + T0 + c + 1;
            while (cover < need) {
                unsigned m = 0xFFFFFFFFu;
#pragma unroll
                for (int i = 0; i < 4; ++i) m = min(min(m, min(r[i][0], r[i][1])), min(r[i][2], r[i][3]));
                if (__builtin_amdgcn_ballot_w64(m == 0) == 0) break;
                cover = avail;
                const unsigned fv = ld_sys(flagp);
                reload();
                avail = rfl(fv);
                if (expired()) cover = avail = 0xFFFFFFFFu;
            }
        };
        settle(t0, 0, [&]() { load_tail(t0, 0); load_tail(t1, 1); });
        compute(t0, T0);
        load_tail(a0, 2);
        settle(t1, 1, [&]() { load_tail(t1, 1); load_tail(a0, 2); });
        compute(t1, T0 + 1);
        load_tail(a1, 3);
        settle(a0, 2, [&]() { load_tail(a0, 2); load_tail(a1, 3); });
        compute(a0, T0 + 2);
        settle(a1, 3, [&]() { load_tail(a1, 3); });
        const unsigned tv = ld_agent(tmo);
        compute(a1, T0 + 3);
        epilogue(1, src * MLOC + unit * 32 + half * 16, bad || rfl(tv) == call);
    }
}

}

extern "C" void kernel_launch(int rank, int nranks, ncclComm_t comm, hipStream_t stream,
                              void* const* d_in, const int* in_sizes, int n_in,
                              void* d_out, int out_size,
                              const int* in_dtypes, int out_dtype,
                              void* const* d_out_all,
                              void* const* d_ws_all, size_t ws_bytes) {
    (void)comm; (void)in_dtypes; (void)out_dtype; (void)d_out_all;
    if (nranks != NR || n_in != 4 || (unsigned)rank >= (unsigned)NR || ws_bytes < WS_NEED ||
        in_sizes[0] != (int)SHARD || in_sizes[1] != KDIM * NLOC || out_size != NR * MLOC * NLOC) {
        fprintf(stderr, "unexpected problem shape\n");
        return;
    }
    Params p;
    p.x = (const unsigned char*)d_in[0];
    p.w = (const unsigned char*)d_in[1];
    p.sx = (const float*)d_in[2];
    p.sw = (const float*)d_in[3];
    p.out = (float*)d_out;
    p.rank = rank;
    {
        int dev = -1;
        const bool ident = hipGetDevice(&dev) == hipSuccess && dev == rank;
        const unsigned grp = 0xB2u;
        unsigned t = (unsigned)rank;
        int l = 1;
        for (int pass = 0; pass < 2; ++pass)
            for (int i = 1; i < NR; ++i) {
                const int q = (rank + i) & 7;
                const bool same = !ident || ((grp >> q) & 1) == ((grp >> rank) & 1);
                if (same == (pass == 0)) t |= (unsigned)q << (4 * l++);
            }
        p.peers = t;
        for (int i = 0; i < NR; ++i) p.ws[i] = (unsigned char*)d_ws_all[(t >> (4 * i)) & 7];
    }
    hipLaunchKernelGGL(ag_gemm_silu_kernel, dim3(256), dim3(512), 0, stream, p);
    HIP_CHECK(hipGetLastError());
}
```
